```python
import numpy as np

kernel_name = "gpu_a2a_gemm_m4096_k4096_n4096_bf16_reluqe4m3"


ABSMAX_RTOL = 0.09

COMPUTE_DTYPE = "bfloat16"
EPILOGUE = "relu_quant_e4m3"
IN_QUANT = None
OUT_QUANT = "e4m3"
IN_DTYPES = {"x": "bfloat16", "w_mat": "bfloat16"}
WORLD_SIZE = 8

def _to_bf16(x):
    u = np.ascontiguousarray(x, dtype=np.float32).view(np.uint32)
    return ((u + 0x7FFF + ((u >> 16) & 1)) & 0xFFFF0000).view(np.float32)

def setup_inputs():

    x = _to_bf16(np.random.randn(4096, 4096).astype(np.float32))
    w_mat = _to_bf16((np.random.randn(4096, 4096) / np.sqrt(4096)).astype(np.float32))
    return {"x": x, "w_mat": w_mat}

def _snap_fp8(x, mant_bits, max_abs, min_normal):
    a = np.abs(x.astype(np.float64))
    _m, _e = np.frexp(a)
    q = np.where(a >= min_normal, np.ldexp(1.0, _e - 1 - mant_bits),
                 min_normal * 2.0 ** -mant_bits)
    snapped = np.minimum(np.round(a / q) * q, max_abs)
    return (np.sign(x) * snapped).astype(np.float32)
def _quant_dequant(y, amax):
    scale = amax / 448.0
    q = _snap_fp8(y / scale, 3, 448.0, 2.0 ** -6)
    return (q * scale).astype(np.float32)
def _epilogue(y):
    y = np.maximum(y, 0.0)
    return _quant_dequant(y, np.abs(y).max())

def reference(x, w_mat):
    y = (x @ w_mat).astype(np.float32)
    return _epilogue(y)
```

```cpp
#include <hip/hip_runtime.h>
#include <rccl/rccl.h>
#include <cstdio>
#include <cstdint>

typedef unsigned u32x4 __attribute__((ext_vector_type(4)));
typedef unsigned u32x2 __attribute__((ext_vector_type(2)));
typedef __bf16 bf16x4 __attribute__((ext_vector_type(4)));
typedef __bf16 bf16x8 __attribute__((ext_vector_type(8)));
typedef float f32x4 __attribute__((ext_vector_type(4)));
typedef unsigned long long u64;
#define LDSP __attribute__((address_space(3)))
#define GLB __attribute__((address_space(1)))
#define DI __device__ __forceinline__

constexpr int NRANK = 8;
constexpr int NWG = 256;
constexpr int NDIM = 4096;
constexpr unsigned XROW = 512 * 2;
constexpr unsigned WROW = NDIM * 2;
constexpr unsigned X_BYTES = 4096u * XROW;
constexpr unsigned W_BYTES = 4096u * WROW;

constexpr size_t WS_CNT = 0;
constexpr size_t WS_TMO = 4096;
constexpr size_t WS_FLAG = 8192;
constexpr size_t WS_GRAN = 16384;
constexpr size_t WS_DBG = 65536;
constexpr size_t WS_STAGE = 1u << 20;
constexpr unsigned STAGE_BYTES = 4u << 20;
constexpr size_t WS_NEED = WS_STAGE + STAGE_BYTES;

constexpr unsigned RING_SLOT = 12288, RING_WAVE = 3 * RING_SLOT;
constexpr unsigned L_CTRL = 4 * RING_WAVE;
constexpr unsigned L_PROG = L_CTRL;
constexpr unsigned L_ABORT = L_CTRL + 32;
constexpr unsigned L_CALL = L_CTRL + 36;
constexpr unsigned L_WMAX = L_CTRL + 48;
constexpr unsigned L_GMAX = L_CTRL + 64;
constexpr unsigned L_BAD = L_CTRL + 80;
constexpr unsigned LDS_BYTES = L_CTRL + 128;

constexpr u64 DEADLINE_TICKS = 50000000ull;

struct Params {
  const void* x;
  const void* w;
  void* out;
  char* ws[NRANK];
  int rank;
  int pad;
};

#define SEND_WIN 3

#define DMA_ON true
#define A_BITS "sc0 sc1"
#define STAMP(slot) do {} while (0)

DI unsigned rfl(unsigned v) { return __builtin_amdgcn_readfirstlane(v); }
DI u32x4 mk_rsrc(const void* ptr, unsigned bytes) {
  u64 a = (u64)ptr;
  u32x4 r = {(unsigned)a, (unsigned)(a >> 32) & 0xffffu, bytes, 0x00020000u};
  return r;
}
DI unsigned lds_ld(unsigned a) { return *(volatile LDSP unsigned*)(size_t)a; }
DI void lds_st(unsigned a, unsigned v) { *(volatile LDSP unsigned*)(size_t)a = v; }
#define SB() __builtin_amdgcn_sched_barrier(0)
#define BAR() do { asm volatile("s_waitcnt lgkmcnt(0)\n\ts_barrier" ::: "memory"); } while (0)

template <int N> DI void wait_vm() { asm volatile("s_waitcnt vmcnt(%0)" ::"i"(N) : "memory"); }
DI void set_m0(unsigned v) { asm volatile("s_mov_b32 m0, %0\n\ts_nop 0" ::"s"(v) : "memory"); }
template <int IMM, bool SYS>
DI void glds(unsigned voff, u32x4 rsrc, unsigned soff) {
  if constexpr (SYS)
    asm volatile("buffer_load_dwordx4 %0, %1, %2 offen offset:%3 " A_BITS " lds" ::"v"(voff), "s"(rsrc), "s"(soff), "i"(IMM) : "memory");
  else
    asm volatile("buffer_load_dwordx4 %0, %1, %2 offen offset:%3 lds" ::"v"(voff), "s"(rsrc), "s"(soff), "i"(IMM) : "memory");
}
template <int IMM, bool SYS>
DI void glds_dep(unsigned voff, u32x4 rsrc, unsigned soff, bf16x8 dep) {
  if constexpr (SYS)
    asm volatile("buffer_load_dwordx4 %0, %1, %2 offen offset:%3 " A_BITS " lds ; %4" ::"v"(voff), "s"(rsrc), "s"(soff), "i"(IMM), "v"(dep) : "memory");
  else
    asm volatile("buffer_load_dwordx4 %0, %1, %2 offen offset:%3 lds ; %4" ::"v"(voff), "s"(rsrc), "s"(soff), "i"(IMM), "v"(dep) : "memory");
}

constexpr int idxA(int i) { return i < 4 ? 2 * i + 1 : 4 + i; }
constexpr int idxB(int j) { return 2 * j; }
constexpr int cntI(int i) { return i <= 4 ? 2 * i : 4 + i; }

DI f32x4 MFMA(bf16x8 a, bf16x8 b, f32x4 c) { return __builtin_amdgcn_mfma_f32_16x16x32_bf16(a, b, c, 0, 0, 0); }
struct Gemm {
  f32x4 acc[8][4];
  bf16x8 Bf[2][4];
  bf16x8 Af[4];
  unsigned vA[8], vB[4];
  unsigned vAl;
  u32x4 rsA, rsW, rsX;
  unsigned ldA_cur, ldA_nxt, ldB_nxt[2];
  unsigned slot_cur;
  unsigned ring0;
  unsigned sA, sW;
  unsigned pi, f;
  unsigned prog;
  unsigned rank, mb, w;
};

DI bf16x8 rdA(unsigned base, int off) { return *(LDSP bf16x8*)(size_t)(base + off); }
DI bf16x8 rdB(unsigned base, int off) {
  bf16x4 lo = __builtin_amdgcn_ds_read_tr16_b64_v4bf16((LDSP bf16x4*)(size_t)(base + off));
  bf16x4 hi = __builtin_amdgcn_ds_read_tr16_b64_v4bf16((LDSP bf16x4*)(size_t)(base + off + 512));
  return __builtin_shufflevector(lo, hi, 0, 1, 2, 3, 4, 5, 6, 7);
}
DI void rdBall(Gemm& g, bf16x8 (&B)[4]) {
  B[0] = rdB(g.ldB_nxt[0], 0);
  B[1] = rdB(g.ldB_nxt[0], 8);
  B[2] = rdB(g.ldB_nxt[1], 0);
  B[3] = rdB(g.ldB_nxt[1], 8);
}

template <int I, bool LOCAL, bool DEP>
DI void issue_iter(Gemm& g, unsigned slot, bf16x8 dep) {
  if constexpr (I == 0) set_m0(slot);
  if constexpr (I == 2) set_m0(slot + 4096);
  if constexpr (I == 4) set_m0(slot + 8192);
  if constexpr (I < 4) {
    if constexpr (DEP) glds_dep<(idxB(I) & 3) * 1024, false>(g.vB[I], g.rsW, g.sW, dep);
    else glds<(idxB(I) & 3) * 1024, false>(g.vB[I], g.rsW, g.sW);
  }
  constexpr int IMM = (idxA(I) & 3) * 1024;
  if constexpr (LOCAL) {
    unsigned so = g.sA + (((I + 2 * g.w) & 7) * 16 * XROW) + 4096 - IMM;
    if constexpr (DEP) glds_dep<IMM, false>(g.vAl, g.rsX, so, dep);
    else glds<IMM, false>(g.vAl, g.rsX, so);
  } else {
    if constexpr (DEP) glds_dep<IMM, true>(g.vA[I], g.rsA, g.sA, dep);
    else glds<IMM, true>(g.vA[I], g.rsA, g.sA);
  }
}

DI void next_remote(Gemm& g) {
  g.pi += 4;
  if (g.pi >= 7) { g.pi -= 7; g.f += 1; }
}
DI void set_remote(Gemm& g) {
  unsigned src = (g.rank + 1 + g.pi) & 7;
  g.sA = src * (512u << 10) + g.f * (32u << 10) + g.mb * (8u << 10);
  g.sW = (src * 512 + g.f * 32) * WROW;
}
DI void set_local(Gemm& g, unsigned t) {
  unsigned f = 4 * t + g.w;
  g.sA = (g.rank * 512 + g.mb * 128) * XROW + f * 64;
  g.sW = (g.rank * 512 + f * 32) * WROW;
}

template <int PAR, int ISSUE, int G, bool NEXT, bool CHK>
DI void step(Gemm& g) {
  if constexpr (ISSUE == 1) {
    unsigned pr = rfl(g.prog);
    while (__builtin_expect(pr <= g.f, 0)) {
      pr = rfl(lds_ld(L_PROG + 4 * g.pi));
      if (rfl(lds_ld(L_ABORT))) { g.rsA[2] = 0; break; }
    }
    set_remote(g);
  }
  const unsigned slot = g.slot_cur;
#define ITER(I)                                                                                                     \
  {                                                                                                                 \
    constexpr int iss = 12 * G + (ISSUE ? cntI(I) : 0);                                                             \
    if constexpr (I >= 1 && I <= 5) wait_vm<iss - idxA(I + 2) - 1>();                                               \
    if constexpr (I == 6 && NEXT) wait_vm<iss - 12 - idxB(3) - 1>();                                                \
    if constexpr (I + 2 < 8) g.Af[(I + 2) & 3] = rdA(g.ldA_cur, idxA(I + 2) * 1024);                                \
    else if constexpr (NEXT) {                                                                                      \
      if constexpr (I == 6 && CHK) { next_remote(g); g.prog = lds_ld(L_PROG + 4 * g.pi); }                          \
      g.Af[(I + 2) & 3] = rdA(g.ldA_nxt, idxA(I - 6) * 1024);                                                       \
      if constexpr (I == 6) rdBall(g, g.Bf[PAR ^ 1]);                                                               \
    }                                                                                                               \
    SB();                                                                                                           \
    if constexpr (ISSUE != 0 && I == 0)                         \
      asm volatile("" ::"v"(g.Bf[PAR][0]), "v"(g.Bf[PAR][1]), "v"(g.Bf[PAR][2]), "v"(g.Bf[PAR][3]) : "memory");     \
    if constexpr (ISSUE != 0 && DMA_ON) issue_iter<I, ISSUE == 2, true>(g, slot, g.Af[I & 3]);                                \
    SB();                                                                                                           \
    _Pragma("unroll") for (int n = 0; n < 4; ++n)                                                                   \
        g.acc[I][n] = MFMA(g.Bf[PAR][n], g.Af[I & 3], g.acc[I][n]);                                                 \
    SB();                                                                                                           \
  }
  ITER(0) ITER(1) ITER(2) ITER(3) ITER(4) ITER(5) ITER(6) ITER(7)
#undef ITER
  if constexpr (NEXT) {
    unsigned nxt = slot + RING_SLOT;
    if (nxt == g.ring0 + RING_WAVE) nxt = g.ring0;
    unsigned n2 = nxt + RING_SLOT;
    if (n2 == g.ring0 + RING_WAVE) n2 = g.ring0;
    g.slot_cur = nxt;
    g.ldA_cur = g.ldA_nxt;
    unsigned d = n2 - nxt;
    g.ldA_nxt += d;
    g.ldB_nxt[0] += d;
    g.ldB_nxt[1] += d;
  }
}

DI unsigned wave_max_u32(unsigned v) {
#pragma unroll
  for (int o = 32; o >= 1; o >>= 1) {
    unsigned t = __shfl_xor(v, o);
    v = v > t ? v : t;
  }
  return v;
}

__global__ void __launch_bounds__(384, 2) a2a_gemm_kernel(Params p) {
  const unsigned tid = threadIdx.x, lane = tid & 63, w = rfl(tid >> 6);
  const unsigned b = blockIdx.x;
  const unsigned rank = p.rank;
  const unsigned mb = (b >> 3) & 3, nt = (b & 7) * 8 + (b >> 5);
  char* const myws = p.ws[rank];
  GLB unsigned* const tmo = (GLB unsigned*)(myws + WS_TMO);

  if (w == 4) {
    STAMP(16);
    const bool on = b < 224;
    const unsigned pi = b % 7, rg = on ? b / 7 : 0, peer = (rank + 1 + pi) & 7;
    u32x4 v[16];
    const GLB char* src = (const GLB char*)p.x + (size_t)(peer * 512 + rg * 16 + (lane & 15)) * XROW + (lane >> 4) * 16;
    if (on) {
#pragma unroll
      for (int f = 0; f < 16; ++f) v[f] = *(const GLB u32x4*)(src + f * 64);
    }
    asm volatile("s_barrier" ::: "memory");
    if (on) {
      const unsigned base = (rfl(lds_ld(L_CALL)) - 1) * 16;
      char* pws = p.ws[peer];
      GLB char* dst = (GLB char*)pws + WS_STAGE + rank * (512u << 10) + rg * 1024 + lane * 16;
      GLB unsigned* flag = (GLB unsigned*)(pws + WS_FLAG + (rank * 4 + (rg >> 3)) * 128 + (rg & 7) * 4);
      asm volatile("" : "+v"(v[0]), "+v"(v[1]), "+v"(v[2]), "+v"(v[3]), "+v"(v[4]), "+v"(v[5]), "+v"(v[6]), "+v"(v[7]));
      asm volatile("" : "+v"(v[8]), "+v"(v[9]), "+v"(v[10]), "+v"(v[11]), "+v"(v[12]), "+v"(v[13]), "+v"(v[14]), "+v"(v[15]));
      asm volatile("s_waitcnt vmcnt(0)" ::: "memory");
      STAMP(17);
      constexpr int WIN = SEND_WIN;
#define ST(F) asm volatile("global_store_dwordx4 %0, %1, off sc0 sc1\n\ts_nop 1" ::"v"(dst + (size_t)(F) * 32768), "v"(v[F]) : "memory")
#pragma unroll
      for (int f = 0; f < WIN; ++f) ST(f);
#pragma unroll
      for (int f = 0; f < 16; ++f) {
        const int s_after = (f + WIN - 1 < 15 ? f + WIN - 1 : 15) - f;
        const int f_after = f < WIN - 1 ? f : WIN - 1;
        asm volatile("s_waitcnt vmcnt(%0)" ::"i"(s_after + f_after) : "memory");
        if (lane == 0) __hip_atomic_store(flag, base + f + 1, __ATOMIC_RELAXED, __HIP_MEMORY_SCOPE_SYSTEM);
        if (f + WIN < 16) ST(f + WIN);
      }
#undef ST
      STAMP(18);
    }
    return;
  }

  if (w == 5) {
    STAMP(20);
    GLB unsigned* cnt = (GLB unsigned*)(myws + WS_CNT) + b;
    const unsigned call = rfl(__hip_atomic_load(cnt, __ATOMIC_RELAXED, __HIP_MEMORY_SCOPE_AGENT)) + 1;
    if (lane < 8) lds_st(L_PROG + 4 * lane, 0);
    if (lane == 8) lds_st(L_ABORT, 0);
    if (lane == 9) lds_st(L_CALL, call);
    BAR();
    if (lane == 0) __hip_atomic_store(cnt, call, __ATOMIC_RELAXED, __HIP_MEMORY_SCOPE_AGENT);
    const unsigned base = (call - 1) * 16;
    const unsigned si = lane >> 3, src = (rank + 1 + si) & 7;
    GLB unsigned* ptr = lane < 56 ? (GLB unsigned*)(myws + WS_FLAG + (src * 4 + mb) * 128 + (lane & 7) * 4) : tmo;
    const u64 t0 = __builtin_amdgcn_s_memrealtime();
    bool first = true;
    for (;;) {
      unsigned v = __hip_atomic_load(ptr, __ATOMIC_RELAXED, __HIP_MEMORY_SCOPE_SYSTEM);
      const bool dead = __any(lane >= 56 && v == call);
      unsigned r = lane < 56 ? v - base : 16;
      unsigned t;
      t = __shfl_xor(r, 1); r = r < t ? r : t;
      t = __shfl_xor(r, 2); r = r < t ? r : t;
      t = __shfl_xor(r, 4); r = r < t ? r : t;
      if ((lane & 7) == 0 && lane < 56)
        __hip_atomic_store((LDSP unsigned*)(size_t)(L_PROG + 4 * si), r, __ATOMIC_RELEASE, __HIP_MEMORY_SCOPE_WORKGROUP);
      if (__all(r >= 16)) break;
      if (dead || __builtin_amdgcn_s_memrealtime() - t0 > DEADLINE_TICKS) {
        if (lane == 0) {
          __hip_atomic_store(tmo, call, __ATOMIC_RELAXED, __HIP_MEMORY_SCOPE_AGENT);
          __hip_atomic_store((LDSP unsigned*)(size_t)L_ABORT, 1u, __ATOMIC_RELEASE, __HIP_MEMORY_SCOPE_WORKGROUP);
        }
        break;
      }
    }
    STAMP(22);
    return;
  }

  STAMP(w);
  Gemm g;
  g.rank = rank; g.mb = mb; g.w = w;
  g.rsA = mk_rsrc(myws + WS_STAGE - 4096, STAGE_BYTES + 4096);
  g.rsW = mk_rsrc(p.w, W_BYTES);
  g.rsX = mk_rsrc((const char*)p.x - 4096, X_BYTES + 4096);
  {
    const unsigned r8 = lane >> 3, c = lane & 7;
#pragma unroll
    for (int j = 0; j < 4; ++j)
      g.vB[j] = (8 * j + r8) * WROW + nt * 128 + ((c ^ ((r8 >> 1) & 1)) << 4) - (idxB(j) & 3) * 1024;
#pragma unroll
    for (int i = 0; i < 8; ++i)
      g.vA[i] = lane * 16 + ((i + 2 * w) & 7) * 1024 + 4096 - (idxA(i) & 3) * 1024;
    g.vAl = (lane & 15) * XROW + (lane >> 4) * 16;
  }
  g.ring0 = w * RING_WAVE;
  {
    bf16x8 z = {};
#define PRO(T)                                                                                         \
  set_local(g, T);                                                                                     \
  issue_iter<0, true, false>(g, g.ring0 + T * RING_SLOT, z); issue_iter<1, true, false>(g, g.ring0 + T * RING_SLOT, z); \
  issue_iter<2, true, false>(g, g.ring0 + T * RING_SLOT, z); issue_iter<3, true, false>(g, g.ring0 + T * RING_SLOT, z); \
  issue_iter<4, true, false>(g, g.ring0 + T * RING_SLOT, z); issue_iter<5, true, false>(g, g.ring0 + T * RING_SLOT, z); \
  issue_iter<6, true, false>(g, g.ring0 + T * RING_SLOT, z); issue_iter<7, true, false>(g, g.ring0 + T * RING_SLOT, z);
    PRO(0) PRO(1) PRO(2)
#undef PRO
  }
#pragma unroll
  for (int i = 0; i < 8; ++i)
#pragma unroll
    for (int n = 0; n < 4; ++n) g.acc[i][n] = f32x4{0.f, 0.f, 0.f, 0.f};
  {
    const unsigned gq = lane >> 4, q = (lane >> 2) & 3, pp = lane & 3;
    const unsigned bl = gq * 2048 + q * 128;
    g.ldB_nxt[0] = g.ring0 + bl + (((2 * pp) ^ ((q >> 1) & 1)) << 4);
    g.ldB_nxt[1] = g.ring0 + bl + (((2 * pp + 1) ^ ((q >> 1) & 1)) << 4);
    g.ldA_cur = g.ring0 + lane * 16;
  }
  asm volatile("s_barrier" ::: "memory");
  wait_vm<36 - idxB(3) - 1>();
  STAMP(4 + w);
  g.Af[0] = rdA(g.ldA_cur, idxA(0) * 1024);
  rdBall(g, g.Bf[0]);
  g.Af[1] = rdA(g.ldA_cur, idxA(1) * 1024);
  g.slot_cur = g.ring0;
  g.ldA_nxt = g.ldA_cur + RING_SLOT;
  g.ldB_nxt[0] += RING_SLOT;
  g.ldB_nxt[1] += RING_SLOT;
  set_local(g, 3);
  g.pi = w + 3; g.f = 0xffffffffu;
  step<0, 2, 3, true, true>(g);
#pragma unroll 1
  for (int it = 0; it < 14; ++it) {
    step<1, 1, 3, true, true>(g);
    step<0, 1, 3, true, true>(g);
  }
  step<1, 0, 3, true, false>(g);
  step<0, 0, 2, true, false>(g);
  step<1, 0, 1, false, false>(g);
  const bool ok = g.rsA[2] != 0;
  STAMP(8 + w);

  {
    const unsigned wr = g.ring0 + lane * 16;
#pragma unroll
    for (int i = 2; i < 8; ++i)
#pragma unroll
      for (int n = 0; n < 4; ++n) *(LDSP f32x4*)(size_t)(wr + ((i - 2) * 4 + n) * 1024) = g.acc[i][n];
    BAR();
#pragma unroll
    for (int q = 1; q < 4; ++q) {
      const unsigned rd = ((w - q) & 3) * RING_WAVE + (q - 1) * 8192 + lane * 16;
#pragma unroll
      for (int h = 0; h < 2; ++h)
#pragma unroll
        for (int n = 0; n < 4; ++n) g.acc[h][n] += *(LDSP f32x4*)(size_t)(rd + (h * 4 + n) * 1024);
    }
  }
  float mx = 0.f;
#pragma unroll
  for (int h = 0; h < 2; ++h)
#pragma unroll
    for (int n = 0; n < 4; ++n)
#pragma unroll
      for (int e = 0; e < 4; ++e) {
        g.acc[h][n][e] = fmaxf(g.acc[h][n][e], 0.f);
        mx = fmaxf(mx, g.acc[h][n][e]);
      }
  unsigned mu = wave_max_u32(__builtin_bit_cast(unsigned, mx));
  if (lane == 0) lds_st(L_WMAX + 4 * w, mu);
  BAR();
  const unsigned call = rfl(lds_ld(L_CALL));
  if (w == 0) {
    u32x4 m4 = *(volatile LDSP u32x4*)(size_t)L_WMAX;
    unsigned m = max(max(m4[0], m4[1]), max(m4[2], m4[3]));
    if (lane < NRANK) {
      char* pw = p.ws[0];
#pragma unroll
      for (int r = 1; r < NRANK; ++r) pw = lane == r ? p.ws[r] : pw;
      GLB u64* gp = (GLB u64*)(pw + WS_GRAN) + rank * NWG + b;
      __hip_atomic_store(gp, ((u64)call << 32) | m, __ATOMIC_RELAXED, __HIP_MEMORY_SCOPE_SYSTEM);
    }
  }
  STAMP(12 + w);
  {
    const GLB char* gp = (const GLB char*)myws + WS_GRAN + w * 4096 + lane * 16;
    const u64 t0 = __builtin_amdgcn_s_memrealtime();
    unsigned gm = 0, bad = 0;
    for (;;) {
      u32x4 a0, a1, a2, a3;
      unsigned tv;
      asm volatile(
          "global_load_dwordx4 %0, %5, off sc0 sc1\n\t"
          "global_load_dwordx4 %1, %5, off offset:1024 sc0 sc1\n\t"
          "global_load_dwordx4 %2, %5, off offset:2048 sc0 sc1\n\t"
          "global_load_dwordx4 %3, %5, off offset:3072 sc0 sc1\n\t"
          "global_load_dword %4, %6, off sc1\n\t"
          "s_waitcnt vmcnt(0)"
          : "=&v"(a0), "=&v"(a1), "=&v"(a2), "=&v"(a3), "=&v"(tv)
          : "v"(gp), "v"(tmo)
          : "memory");
      const bool all = a0[1] == call && a0[3] == call && a1[1] == call && a1[3] == call && a2[1] == call &&
                       a2[3] == call && a3[1] == call && a3[3] == call;
      if (rfl(tv) == call) { bad = 1; break; }
      if (__all(all)) {
        gm = max(max(max(a0[0], a0[2]), max(a1[0], a1[2])), max(max(a2[0], a2[2]), max(a3[0], a3[2])));
        break;
      }
      if (__builtin_amdgcn_s_memrealtime() - t0 > DEADLINE_TICKS) {
        if (lane == 0) __hip_atomic_store(tmo, call, __ATOMIC_RELAXED, __HIP_MEMORY_SCOPE_AGENT);
        bad = 1;
        break;
      }
    }
    if (!ok) bad = 1;
    gm = wave_max_u32(gm);
    if (lane == 0) { lds_st(L_GMAX + 4 * w, gm); lds_st(L_BAD + 4 * w, bad); }
  }
  BAR();
  STAMP(24 + w);
  {
    u32x4 m4 = *(volatile LDSP u32x4*)(size_t)L_GMAX;
    u32x4 b4 = *(volatile LDSP u32x4*)(size_t)L_BAD;
    const float amax = __builtin_bit_cast(float, max(max(m4[0], m4[1]), max(m4[2], m4[3])));
    const bool bad = (b4[0] | b4[1] | b4[2] | b4[3]) != 0;
    const float s = amax / 448.0f;
    const float inv = 1.0f / s;
#pragma unroll
    for (int h = 0; h < 2; ++h) {
      u32x4 o;
#pragma unroll
      for (int n = 0; n < 4; ++n) {
        f32x4 y = g.acc[h][n];
        int q = __builtin_amdgcn_cvt_pk_fp8_f32(fminf(y[0] * inv, 448.f), fminf(y[1] * inv, 448.f), 0, false);
        q = __builtin_amdgcn_cvt_pk_fp8_f32(fminf(y[2] * inv, 448.f), fminf(y[3] * inv, 448.f), q, true);
        o[n] = bad ? 0x7f7f7f7fu : (unsigned)q;
      }
      GLB char* op = (GLB char*)p.out + (size_t)(mb * 128 + (2 * w + h) * 16 + (lane & 15)) * NDIM + nt * 64 + (lane >> 4) * 16;
      *(GLB u32x4*)op = o;
    }
  }
  STAMP(28 + w);
}

#define HIP_CHECK(expr) do { hipError_t _e = (expr); if (_e != hipSuccess) { \
    fprintf(stderr, "HIP error %s at %s:%d\n", hipGetErrorString(_e), __FILE__, __LINE__); \
    return; } } while (0)

extern "C" void kernel_launch(int rank, int nranks, ncclComm_t comm, hipStream_t stream, void* const* d_in,
                              const int* in_sizes, int n_in, void* d_out, int out_size, const int* in_dtypes,
                              int out_dtype, void* const* d_out_all, void* const* d_ws_all, size_t ws_bytes) {
  if (nranks != NRANK || (unsigned)rank >= (unsigned)NRANK || ws_bytes < WS_NEED || n_in < 2 ||
      in_sizes[0] != 4096 * 512 || in_sizes[1] != 4096 * 4096 || out_size != 512 * 4096) {
    fprintf(stderr, "unexpected problem shape\n");
    return;
  }
  HIP_CHECK(hipFuncSetAttribute((const void*)a2a_gemm_kernel, hipFuncAttributeMaxDynamicSharedMemorySize, LDS_BYTES));
  Params p;
  p.x = d_in[0];
  p.w = d_in[1];
  p.out = d_out;
  for (int r = 0; r < NRANK; ++r) p.ws[r] = (char*)d_ws_all[r];
  p.rank = rank;
  p.pad = 0;
  hipLaunchKernelGGL(a2a_gemm_kernel, dim3(NWG), dim3(384), LDS_BYTES, stream, p);
  HIP_CHECK(hipGetLastError());
}
```

```cpp
#include <hip/hip_runtime.h>
#include <rccl/rccl.h>
#include <cstdio>
#include <cstdint>

typedef unsigned u32x4 __attribute__((ext_vector_type(4)));
typedef unsigned u32x2 __attribute__((ext_vector_type(2)));
typedef __bf16 bf16x4 __attribute__((ext_vector_type(4)));
typedef __bf16 bf16x8 __attribute__((ext_vector_type(8)));
typedef float f32x4 __attribute__((ext_vector_type(4)));
typedef unsigned long long u64;
#define LDSP __attribute__((address_space(3)))
#define GLB __attribute__((address_space(1)))
#define DI __device__ __forceinline__

constexpr int NRANK = 8;
constexpr int NWG = 256;
constexpr int NTHREADS = 704;
constexpr int NDIM = 4096;
constexpr unsigned XROW = 512 * 2;
constexpr unsigned WROW = NDIM * 2;
constexpr unsigned X_BYTES = 4096u * XROW;
constexpr unsigned W_BYTES = 4096u * WROW;
#define F2 14
constexpr unsigned NU1 = 8 + F2 / 2 * 7, NU = NU1 + (16 - F2) * 7;
constexpr unsigned NP2 = (NU - NU1) / 2;
#define XSWZ(row) (((row) >> 3) << 1)

constexpr size_t WS_CNT = 0;
constexpr size_t WS_TMO = 4096;
constexpr size_t WS_G2 = 8192;
constexpr size_t WS_FLAG = 16384;
constexpr size_t WS_XFLAG = 32768;
constexpr size_t WS_G1 = 65536;
constexpr size_t WS_DBG = 131072;
constexpr size_t WS_STAGE = 1u << 20;
constexpr unsigned STAGE_BYTES = 4u << 20;
constexpr size_t WS_XBUF = WS_STAGE + STAGE_BYTES;
constexpr unsigned XBUF_BYTES = 8u << 20;
constexpr size_t WS_NEED = WS_XBUF + XBUF_BYTES;

#define NSLOT_A 9
#define NSLOT_W 10
constexpr unsigned NSA = NSLOT_A, NSW = NSLOT_W, SLOT = 8192;
constexpr unsigned L_W = NSA * SLOT, L_CTRL = L_W + NSW * SLOT;
constexpr unsigned L_FULL = L_CTRL + (NU1 & 1) * 8;
constexpr unsigned L_FREE = L_CTRL + 768;
constexpr unsigned L_RAW = L_CTRL + 1792;
constexpr unsigned L_GO = L_CTRL + 2308;
constexpr unsigned L_WMAX = L_CTRL + 2320;
constexpr unsigned L_RES = L_CTRL + 2336;
constexpr unsigned CTRL_BYTES = 2560;
static_assert(NU + 3 <= 96, "");
constexpr unsigned L_JUNK = L_CTRL + CTRL_BYTES;
constexpr unsigned LDS_BYTES = L_JUNK + 256;

constexpr u64 DEADLINE_TICKS = 50000000ull;

struct Params {
  const void* x;
  const void* w;
  void* out;
  char* peer[NRANK];
  int rank;
  int pad;
};

#define PF_FIRST 7
#define LGAP() do {} while (0)
#define LIM() FENCE();
#define SPIN_SLEEP 1
#define SPIN() do { if (SPIN_SLEEP) __builtin_amdgcn_s_sleep(SPIN_SLEEP); } while (0)
#define PACE 90
#define SEND_T0 0
#define SEND_LATE 16
#define SEND_LD_AT 150
#define GO_LATE
#define NO_PF
#define SEND_ASAP 2
#define SEND_E0 130
#define SEND_EP 40
#define SEND_EARLY 2
#define POLL_AHEAD 1
#define PF_END 2
#define PF_GATE 7
#define PF_BURST 3
#define PF_STAG 3
#define PF_DIST 20
#define POLL_SLEEP 2
#define XAUX 17
#define XFLAG_AT 6
#define XLOAD_AT 7

#define TIC() do {} while (0)
#define TOC(acc) do {} while (0)
#define RAW_OUT(base, a, b_, c_, d) do {} while (0)
#define STAMP(i) do {} while (0)
#define STAMP_DECL do {} while (0)
#define STAMP_OUT(base) do {} while (0)

DI unsigned rfl(unsigned v) { return __builtin_amdgcn_readfirstlane(v); }
DI u32x4 mk_rsrc(const void* ptr, unsigned bytes) {
  u64 a = (u64)ptr;
  u32x4 r = {(unsigned)a, (unsigned)(a >> 32) & 0xffffu, bytes, 0x00020000u};
  return r;
}
DI unsigned lds_ld(unsigned a) { return *(volatile LDSP unsigned*)(size_t)a; }
DI u32x2 lds_ld2(unsigned a) { return *(volatile LDSP u32x2*)(size_t)a; }
DI u32x4 lds_ld4(unsigned a) { return *(volatile LDSP u32x4*)(size_t)a; }
DI void lds_st(unsigned a, unsigned v) { *(volatile LDSP unsigned*)(size_t)a = v; }
DI unsigned min4(u32x4 v) { return min(min(v[0], v[1]), min(v[2], v[3])); }
#define SB() __builtin_amdgcn_sched_barrier(0)
#define FENCE() asm volatile("" ::: "memory")

template <int N> DI void wait_vm() { asm volatile("s_waitcnt vmcnt(%0)" ::"i"(N) : "memory"); }
DI void set_m0(unsigned v) { asm volatile("s_mov_b32 m0, %0\n\ts_nop 0" ::"s"(v) : "memory"); }
DI void set_m0_5(unsigned v) { asm volatile("s_mov_b32 m0, %0\n\ts_nop 3" ::"s"(v) : "memory"); }
template <int IMM, bool SYS>
DI void glds(unsigned voff, u32x4 rsrc, unsigned soff) {
  if constexpr (SYS)
    asm volatile("buffer_load_dwordx4 %0, %1, %2 offen offset:%3 sc0 sc1 lds" ::"v"(voff), "s"(rsrc), "s"(soff), "i"(IMM) : "memory");
  else
    asm volatile("buffer_load_dwordx4 %0, %1, %2 offen offset:%3 lds" ::"v"(voff), "s"(rsrc), "s"(soff), "i"(IMM) : "memory");
}
DI u32x4 mark_rsrc() {
  u64 pc;
  asm volatile("s_getpc_b64 %0\n\ts_nop 0" : "=s"(pc));
  return mk_rsrc((const void*)pc, 4);
}
DI void mark(u32x4 rm, unsigned lds) {
  asm volatile("s_mov_b32 m0, %1\n\ts_mov_b64 exec, 1\n\tbuffer_load_dword off, %0, 0 lds\n\ts_mov_b64 exec, -1" ::"s"(rm), "s"(lds) : "memory");
}

DI u32x4 seq_rsrc() {
  u64 pc;
  asm volatile("s_getpc_b64 %0\n\ts_branch .Lseq%=\n\t.set .Lv%=, 0\n\t.rept 96\n\t.long .Lv%=\n\t.set .Lv%=, .Lv%=+1\n\t.endr\n.Lseq%=:" : "=s"(pc));
  return mk_rsrc((const void*)(pc + 4), 384);
}
DI void count(u32x4 rq, unsigned off) { asm volatile("buffer_load_dword off, %0, %1 lds" ::"s"(rq), "s"(off) : "memory"); }

DI bool give_up(GLB unsigned* tmo, unsigned call, u64 t0, unsigned lane) {
  if (rfl(__hip_atomic_load(tmo, __ATOMIC_RELAXED, __HIP_MEMORY_SCOPE_AGENT)) != call &&
      __builtin_amdgcn_s_memrealtime() - t0 <= DEADLINE_TICKS) return false;
  if (lane == 0) __hip_atomic_store(tmo, call, __ATOMIC_RELAXED, __HIP_MEMORY_SCOPE_AGENT);
  return true;
}

DI unsigned wave_max_u32(unsigned v) {
  v = max(v, (unsigned)__builtin_amdgcn_update_dpp(0, v, 0x111, 0xf, 0xf, true));
  v = max(v, (unsigned)__builtin_amdgcn_update_dpp(0, v, 0x112, 0xf, 0xf, true));
  v = max(v, (unsigned)__builtin_amdgcn_update_dpp(0, v, 0x114, 0xf, 0xf, true));
  v = max(v, (unsigned)__builtin_amdgcn_update_dpp(0, v, 0x118, 0xf, 0xf, true));
  unsigned a = __builtin_amdgcn_readlane(v, 15), b = __builtin_amdgcn_readlane(v, 31);
  unsigned c = __builtin_amdgcn_readlane(v, 47), d = __builtin_amdgcn_readlane(v, 63);
  return max(max(a, b), max(c, d));
}
DI void unit_of(unsigned s, unsigned rank, unsigned kh, unsigned rot, unsigned& src, unsigned& f, unsigned& pi) {
  if (s < 8) { src = rank; f = 2 * ((s + rot) & 7) + kh; pi = 0; return; }
  const unsigned v = s < NU1 ? s - 8 : s - NU1, j = (v * 37) >> 8;
  const unsigned i = v - 7 * j;
  pi = (i + rot) % 7;
  f = s < NU1 ? 2 * j + kh : F2 + j;
  src = (rank + 1 + pi) & 7;
}

DI f32x4 MFMA(bf16x8 a, bf16x8 b, f32x4 c) {
  asm volatile("v_mfma_f32_16x16x32_bf16 %0, %1, %2, %0" : "+v"(c) : "v"(a), "v"(b));
  return c;
}

#define MFMA_DONE() do { SB(); asm volatile("s_nop 7\n\ts_nop 1" ::: "memory"); SB(); } while (0)

struct Cons {
  f32x4 acc[4][4];
  bf16x8 A[2][4], B[2][4], B2[2][4];
  unsigned la[2], lb[4];
  unsigned sa, sw;
  unsigned vfull;
  unsigned nfree;
  unsigned ok;
  u32x4 rq;
  unsigned spin;
  unsigned tv, tn;
};
DI unsigned nxa(unsigned s) { s += SLOT; return s == NSA * SLOT ? 0 : s; }
DI unsigned nxw(unsigned s) { s += SLOT; return s == NSW * SLOT ? 0 : s; }
DI bf16x8 rdA(unsigned a, int off) { return *(LDSP bf16x8*)(size_t)(a + off); }
DI bf16x8 rdB(unsigned a) {
  bf16x4 lo = __builtin_amdgcn_ds_read_tr16_b64_v4bf16((LDSP bf16x4*)(size_t)a);
  bf16x4 hi = __builtin_amdgcn_ds_read_tr16_b64_v4bf16((LDSP bf16x4*)(size_t)(a + 1024));
  return __builtin_shufflevector(lo, hi, 0, 1, 2, 3, 4, 5, 6, 7);
}
template <bool PAIR> DI void need(Cons& c) {
  if (__builtin_expect(c.ok == 0, 0)) {
    TIC();
    if constexpr (PAIR) { while (rfl(min4(lds_ld4(c.vfull))) == 0) {} }
    else { u32x2 f; do { f = lds_ld2(c.vfull); } while (rfl(min(f[0], f[1])) == 0); }
    TOC(c.spin);
  }
  FENCE();
}
#define M(I, N, BB, AI) c.acc[I][N] = MFMA(c.BB[P][N], c.A[P][AI], c.acc[I][N]); SB();
#define PIN(v) asm volatile("" : "+v"(v))
#define PINS(v) asm volatile("" : "+s"(v))
template <int P, int MODE>
DI void unit1(Cons& c) {
  constexpr int Q = P ^ 1;
  const unsigned sa = c.sa, sw = c.sw;
  unsigned t, ra, m;
  u32x4 f4; u32x2 f2;
  M(0, 0, B, 0) need<MODE == 2>(c); t = c.lb[0] + sw; PIN(t); SB();
  M(0, 1, B, 0) c.B[Q][0] = rdB(t); SB();
  M(0, 2, B, 0) ra = c.la[0] + sa; c.A[Q][0] = rdA(ra, 0); SB();
  M(0, 3, B, 0) t = c.lb[1] + sw; PIN(t); c.A[Q][1] = rdA(ra, 1024); SB();
  M(1, 0, B, 1)
                asm volatile("" ::"v"(c.A[P][2]), "v"(c.A[P][3]) : "memory"); count(c.rq, c.nfree); SB();
  M(1, 1, B, 1) c.B[Q][1] = rdB(t); SB();
  M(1, 2, B, 1) t = c.lb[2] + sw; PIN(t); c.sa = sa + SLOT; PINS(c.sa); SB();
  M(1, 3, B, 1) c.B[Q][2] = rdB(t); SB();
  M(2, 0, B, 2) t = c.lb[3] + sw; PIN(t); c.vfull += MODE == 2 ? 16 : 8; PIN(c.vfull); SB();
  M(2, 1, B, 2) c.B[Q][3] = rdB(t); SB();
  M(2, 2, B, 2) c.sa = c.sa == NSA * SLOT ? 0 : c.sa; PINS(c.sa); SB();
  M(2, 3, B, 2) ra = MODE == 2 ? c.la[0] + c.sa : c.la[1] + sa;
                if constexpr (MODE == 1) f4 = lds_ld4(c.vfull); else if constexpr (MODE == 0) f2 = lds_ld2(c.vfull);
                SB();
  M(3, 0, B, 3) c.A[Q][2] = rdA(ra, 0); c.A[Q][3] = rdA(ra, 1024); SB();
  M(3, 1, B, 3) c.sw = sw + SLOT; c.nfree += 4; PINS(c.sw); PINS(c.nfree); SB();
  M(3, 2, B, 3) c.sw = c.sw == NSW * SLOT ? 0 : c.sw; PINS(c.sw); SB();
  M(3, 3, B, 3) if constexpr (MODE == 1) c.ok = rfl(min4(f4)); else if constexpr (MODE == 0) c.ok = rfl(min(f2[0], f2[1]));
                PINS(c.ok); SB();
}
template <int P, bool NEXT>
DI void unit2(Cons& c) {
  constexpr int Q = P ^ 1;
  const unsigned sa = c.sa, sw = c.sw;
  unsigned t, ra, sa1, sw1;
  u32x4 f4;
  M(0, 0, B, 0) if constexpr (NEXT) { need<true>(c); t = c.lb[0] + sw; PIN(t); } SB();
  M(0, 1, B, 0) if constexpr (NEXT) { c.B[Q][0] = rdB(t); t = c.lb[1] + sw; PIN(t); } SB();
  M(0, 2, B, 0) if constexpr (NEXT) { ra = c.la[0] + sa; c.A[Q][0] = rdA(ra, 0); c.A[Q][1] = rdA(ra, 1024); } SB();
  M(0, 3, B, 0) if constexpr (NEXT) { c.B[Q][1] = rdB(t); t = c.lb[2] + sw; PIN(t); } SB();
  M(1, 0, B, 1) if constexpr (NEXT) {
                  asm volatile("" ::"v"(c.A[P][3]), "v"(c.B2[P][0]), "v"(c.B2[P][1]), "v"(c.B2[P][2]), "v"(c.B2[P][3]) : "memory");
                  count(c.rq, c.nfree); } SB();
  M(1, 1, B, 1) if constexpr (NEXT) { c.B[Q][2] = rdB(t); t = c.lb[3] + sw; PIN(t); } SB();
  M(1, 2, B, 1) if constexpr (NEXT) { sw1 = nxw(sw); PINS(sw1); } SB();
  M(1, 3, B, 1) if constexpr (NEXT) { c.B[Q][3] = rdB(t); t = c.lb[0] + sw1; PIN(t); } SB();
  M(0, 0, B2, 2) if constexpr (NEXT) { sa1 = nxa(sa); PINS(sa1); } SB();
  M(0, 1, B2, 2) if constexpr (NEXT) { c.B2[Q][0] = rdB(t); t = c.lb[1] + sw1; PIN(t); } SB();
  M(0, 2, B2, 2) if constexpr (NEXT) { ra = c.la[0] + sa1; c.A[Q][2] = rdA(ra, 0); c.vfull += 16; PIN(c.vfull); } SB();
  M(0, 3, B2, 2) if constexpr (NEXT) { c.B2[Q][1] = rdB(t); t = c.lb[2] + sw1; PIN(t); } SB();
  M(1, 0, B2, 3) if constexpr (NEXT) { f4 = lds_ld4(c.vfull); c.B2[Q][2] = rdB(t); t = c.lb[3] + sw1; PIN(t); } SB();
  M(1, 1, B2, 3) if constexpr (NEXT) { c.B2[Q][3] = rdB(t); c.A[Q][3] = rdA(ra, 1024); } SB();
  M(1, 2, B2, 3) if constexpr (NEXT) { c.sa = nxa(sa1); c.nfree += 8; PINS(c.sa); PINS(c.nfree); } SB();
  M(1, 3, B2, 3) if constexpr (NEXT) { c.sw = nxw(sw1); PINS(c.sw); c.ok = rfl(min4(f4)); PINS(c.ok); } SB();
}
#undef M

template <int I> struct ic { static constexpr int value = I; };
template <int N, int I = 0, class F> DI void static_for(F& f) {
  if constexpr (I < N) { f(ic<I>{}); static_for<N, I + 1>(f); }
}
#define TRACE() do {} while (0)
__global__ void __launch_bounds__(NTHREADS, 3) a2a_gemm_kernel(Params p) {
  STAMP_DECL;
  STAMP(0);
  const unsigned tid = threadIdx.x, lane = tid & 63, wave = rfl(tid >> 6);
  const unsigned b = blockIdx.x;
  const unsigned rank = p.rank;
  const unsigned kh = (b >> 2) & 1, mb = (b >> 3) & 3, ct = (b & 3) * 8 + (b >> 5);
#define ROT (2 * mb + (b >> 5))
  const unsigned rot = ROT;
  char* const myws = p.peer[7];
  const unsigned sidx = (b & 7) | ((b >> 4) << 3);
  char* const pws = p.peer[sidx % 7];
  const unsigned sf = (sidx / 14) * 2 + ((b >> 3) & 1);
  asm volatile("" ::"s"(p.x), "s"(p.w), "s"(p.out), "s"(myws), "s"(pws), "s"(rank));
  GLB unsigned* const tmo = (GLB unsigned*)(myws + WS_TMO);
  GLB unsigned* const cnt = (GLB unsigned*)(myws + WS_CNT) + b;

  if (wave == 0) {
    __builtin_amdgcn_s_setprio(3);
    if (b >= 224) {
      asm volatile("s_barrier" ::: "memory");
      return;
    }
    const u64 t0 = __builtin_amdgcn_s_memrealtime();
    const unsigned pi = sidx % 7, half = (sidx / 7) & 1, f = sf, peer = (rank + 1 + pi) & 7;
    u32x4 v[16];
    unsigned c0;
    const unsigned row = lane >> 2;
    const u32x4 rsX = mk_rsrc(p.x, X_BYTES);
    const unsigned vo = row * XROW + (((lane & 3) ^ XSWZ(row)) << 4), so = (peer * 512 + half * 256) * XROW + f * 64;
    if (f >= SEND_LATE) {
      asm volatile("s_barrier" ::: "memory");
      while (__builtin_amdgcn_s_memrealtime() - t0 < SEND_LD_AT) __builtin_amdgcn_s_sleep(2);
    }
#pragma unroll
    for (int i = 0; i < 16; ++i)
      asm volatile("buffer_load_dwordx4 %0, %1, %2, %3 offen" : "=v"(v[i]) : "v"(vo), "s"(rsX), "s"(so + i * 16 * XROW) : "memory");
    asm volatile("global_load_dword %0, %1, off sc1" : "=v"(c0) : "v"(cnt) : "memory");
    STAMP(1);
    if (f < SEND_LATE) asm volatile("s_barrier" ::: "memory");
    asm volatile("s_waitcnt vmcnt(0)" : "+v"(v[0]), "+v"(v[1]), "+v"(v[2]), "+v"(v[3]), "+v"(v[4]), "+v"(v[5]), "+v"(v[6]), "+v"(v[7]), "+v"(c0)::"memory");
    asm volatile("" : "+v"(v[8]), "+v"(v[9]), "+v"(v[10]), "+v"(v[11]), "+v"(v[12]), "+v"(v[13]), "+v"(v[14]), "+v"(v[15]));
    STAMP(2);
    GLB char* dst = (GLB char*)pws + WS_STAGE + rank * (512u << 10) + f * 32768 + half * 16384 + lane * 16;
    GLB unsigned* flag = (GLB unsigned*)(pws + WS_FLAG + (half * 16 + f) * 128) + rank;
    const unsigned due = f < SEND_ASAP ? 0 : f < SEND_EARLY ? SEND_E0 + (2 * f + half - 2 * SEND_ASAP) * SEND_EP / 2 : SEND_T0 + (2 * f + half) * PACE / 2;
    while (__builtin_amdgcn_s_memrealtime() - t0 < due) __builtin_amdgcn_s_sleep(2);
    STAMP(3);
#pragma unroll
    for (int i = 0; i < 16; ++i)
      asm volatile("global_store_dwordx4 %0, %1, off offset:%2 sc0 sc1\n\ts_nop 1" ::"v"(dst + (i >> 2) * 4096), "v"(v[i]), "i"((i & 3) * 1024) : "memory");
    STAMP(4);
    wait_vm<0>();
    if (lane == 0) __hip_atomic_store(flag, c0 + 1, __ATOMIC_RELAXED, __HIP_MEMORY_SCOPE_SYSTEM);
    if (f < SEND_EARLY) lds_st(L_GO, 1);
    STAMP(5);
    STAMP_OUT(0);
    return;
  }

  if (wave == 1) {
    unsigned c0;
    asm volatile("global_load_dword %0, %1, off sc1" : "=v"(c0) : "v"(cnt) : "memory");
#pragma unroll
    for (unsigned i = 0; i < CTRL_BYTES / 256; ++i) lds_st(L_CTRL + 256 * i + 4 * lane, 0);
    asm volatile("s_waitcnt lgkmcnt(0)\n\ts_barrier\n\ts_waitcnt vmcnt(0)" : "+v"(c0)::"memory");
    const unsigned call = rfl(c0) + 1;
    bool dead = false;
    GLB u64* g2 = (GLB u64*)(p.peer[lane & 7] + WS_G2 + (rank * 8 + (b & 7)) * 128);
    asm volatile("" : "+v"(g2));
    STAMP(1);
    u32x4 m4;
    for (;;) { m4 = lds_ld4(L_WMAX); if (rfl(min4(m4))) break; __builtin_amdgcn_s_sleep(2); }
    const u64 t0 = __builtin_amdgcn_s_memrealtime();
    const unsigned wgmax = max(max(m4[0], m4[1]), max(m4[2], m4[3])) - 1;
    if (lane == 0) {
      __hip_atomic_store((GLB u64*)(myws + WS_G1 + b * 128), ((u64)call << 32) | wgmax, __ATOMIC_RELAXED, __HIP_MEMORY_SCOPE_AGENT);
      __hip_atomic_store(cnt, call, __ATOMIC_RELAXED, __HIP_MEMORY_SCOPE_AGENT);
    }
    STAMP(6);
    if (b < 8 && !dead) {
      GLB u64* gp = lane < 32 ? (GLB u64*)(myws + WS_G1 + (b * 32 + lane) * 128) : (GLB u64*)tmo;
      for (;;) {
        const u64 v = __hip_atomic_load(gp, __ATOMIC_RELAXED, __HIP_MEMORY_SCOPE_AGENT);
        if (__all(lane >= 32 || (unsigned)(v >> 32) == call)) {
          const unsigned m = wave_max_u32(lane < 32 ? (unsigned)v : 0u);
          if (lane < 8) __hip_atomic_store(g2, ((u64)call << 32) | m, __ATOMIC_RELAXED, __HIP_MEMORY_SCOPE_SYSTEM);
          break;
        }
        if (__any(lane >= 32 && (unsigned)v == call) || __builtin_amdgcn_s_memrealtime() - t0 > DEADLINE_TICKS) {
          if (lane == 0) __hip_atomic_store(tmo, call, __ATOMIC_RELAXED, __HIP_MEMORY_SCOPE_AGENT);
          dead = true;
          break;
        }
      }
    }
    unsigned gm = 0;
    const GLB char* gp = (const GLB char*)myws + WS_G2 + lane * 128;
    while (!dead) {
      u32x2 a;
      unsigned tv;
      asm volatile(
          "global_load_dwordx2 %0, %2, off sc0 sc1\n\t"
          "global_load_dword %1, %3, off sc1\n\t"
          "s_waitcnt vmcnt(0)"
          : "=&v"(a), "=&v"(tv)
          : "v"(gp), "v"(tmo)
          : "memory");
      if (rfl(tv) == call) { dead = true; break; }
      if (__all(a[1] == call)) { gm = wave_max_u32(a[0]); break; }
      if (__builtin_amdgcn_s_memrealtime() - t0 > DEADLINE_TICKS) {
        if (lane == 0) __hip_atomic_store(tmo, call, __ATOMIC_RELAXED, __HIP_MEMORY_SCOPE_AGENT);
        dead = true;
      }
    }
    if (lane == 0) *(volatile LDSP u32x2*)(size_t)L_RES = u32x2{dead ? 2u : 1u, gm};
    STAMP(7);
    STAMP_OUT(1);
    return;
  }

  if (wave == 2) {
    const u32x4 rsW = mk_rsrc(p.w, W_BYTES);
    set_m0(L_JUNK);
    auto pfw = [&](unsigned j) {
      const unsigned s = min(4 * j + (lane >> 4), NU - 1);
      unsigned src, f, pi;
      unit_of(s, rank, kh, 0, src, f, pi);
      const unsigned off = (src * 512 + f * 32 + mb * 8 + ((lane >> 1) & 7)) * WROW + ct * 256 + (lane & 1) * 128;
      asm volatile("buffer_load_dword %0, %1, 0 offen sc1 lds" ::"v"(off), "s"(rsW) : "memory");
    };
    asm volatile("s_barrier" ::: "memory");
    STAMP(2);
    STAMP_OUT(2);
    return;
  }

  if (wave < 7) {
    const unsigned k = (wave - 3) & 1, isW = wave >= 5;
    const u32x4 rm = mark_rsrc();
    unsigned tabS, tabP;
    {
      unsigned src, f, pi;
      const unsigned s = min(2 * lane + k, NU - 1);
      unit_of(s, rank, kh, rot, src, f, pi);
      tabS = isW ? (src * 16 + f) << 18 : s < 8 ? (rank * 512 + mb * 128) * XROW + f * 64 : ((src * 16 + f) << 15) + (s < NU1 ? 0 : kh * 2048);
      tabP = L_RAW + f * 32 + src * 4;
    }
    const unsigned ns = isW ? NSW : NSA, ring = isW ? L_W : 0;
    unsigned slot = k * SLOT, t = 0;
    unsigned sfull = L_FULL + 8 * k + 4 * isW;
    const unsigned vfree = L_FREE + (lane & 3) * 256;
    unsigned c1 = 0, c2 = 0;
    unsigned fr = 0;
#define HEAD                                                                                       \
  const unsigned so = __builtin_amdgcn_readlane(tabS, t);                                          \
  if (__builtin_expect(__any(fr + ns <= 2 * t + k), 0)) { TIC(); do { SPIN(); fr = lds_ld(vfree); } while (__any(fr + ns <= 2 * t + k)); TOC(c1); } \
  FENCE();
#define MID fr = lds_ld(vfree);
#define NEXT                                                                                       \
  mark(rm, sfull);                                                                                 \
  sfull += 16;                                                                                     \
  slot += 2 * SLOT; if (slot >= ns * SLOT) slot -= ns * SLOT;
    if (isW) {
      u32x4 rs = mk_rsrc((const char*)p.w - 4096, W_BYTES + 4096);
      const unsigned rl = lane >> 4, c = lane & 15;
      const unsigned v0 = rl * WROW + ct * 256 + ((c ^ rl) << 4) + 4096, v1 = v0 ^ 128;
      constexpr unsigned D = 4 * WROW - 1024;
      asm volatile("s_barrier" ::: "memory");
      if (sf < SEND_EARLY) while (rfl(lds_ld(L_GO)) == 0) __builtin_amdgcn_s_sleep(1);
#pragma unroll 1
      for (; 2 * t + k < NU; ++t) {
        HEAD
        set_m0_5(ring + slot);
        glds<0, false>(v0, rs, so); glds<1024, false>(v0, rs, so + D); glds<2048, false>(v1, rs, so + 2 * D); glds<3072, false>(v1, rs, so + 3 * D);
        MID
        set_m0(ring + slot + 4096);
        glds<0, false>(v0, rs, so + 16 * WROW); glds<1024, false>(v0, rs, so + 16 * WROW + D);
        glds<2048, false>(v1, rs, so + 16 * WROW + 2 * D); glds<3072, false>(v1, rs, so + 16 * WROW + 3 * D);
        NEXT
      }
      STAMP(1);
      STAMP_OUT(4);
      return;
    }
    unsigned c0;
    asm volatile("global_load_dword %0, %1, off sc1" : "=v"(c0) : "v"(cnt) : "memory");
    u32x4 rs = mk_rsrc((const char*)p.x - 4096, X_BYTES + 4096);
    const unsigned row = lane >> 2;
    const unsigned vx = row * XROW + (((lane & 3) ^ XSWZ(row)) << 4) + 4096, va = lane * 16;
    asm volatile("s_barrier" ::: "memory");
    if (sf < SEND_EARLY) while (rfl(lds_ld(L_GO)) == 0) __builtin_amdgcn_s_sleep(1);
#pragma unroll 1
    for (; t < 4; ++t) {
      HEAD
      set_m0_5(slot);
      glds<0, false>(vx, rs, so); glds<1024, false>(vx, rs, so + 15360);
      glds<2048, false>(vx, rs, so + 2 * 15360); glds<3072, false>(vx, rs, so + 3 * 15360);
      MID
      set_m0(slot + 4096);
      glds<0, false>(vx, rs, so + 65536); glds<1024, false>(vx, rs, so + 65536 + 15360);
      glds<2048, false>(vx, rs, so + 65536 + 2 * 15360); glds<3072, false>(vx, rs, so + 65536 + 3 * 15360);
      NEXT
    }
    STAMP(1);
    asm volatile("s_waitcnt vmcnt(36)" : "+v"(c0)::"memory");
    const unsigned call = rfl(c0) + 1;
    rs = mk_rsrc(myws + WS_STAGE + mb * 8192, STAGE_BYTES - mb * 8192);
    u64 ready = ~0ull << ((NU + 1 - k) / 2);
    unsigned atr = 0;
    const u32x4 rsF = mk_rsrc(myws + WS_FLAG + (mb >> 1) * 2048, 2048);
    const unsigned vlane = lane * 4;
    auto poll = [&](unsigned i) {
      const unsigned a = __builtin_amdgcn_readlane(tabP, i) & ~31u;
      asm volatile("s_mov_b32 m0, %2\n\ts_mov_b64 exec, 0xff\n\ts_nop 2\n\tbuffer_load_dword %0, %1, %3 offen sc0 sc1 lds\n\ts_mov_b64 exec, -1"
                   ::"v"(vlane), "s"(rsF), "s"(a), "s"((a - L_RAW) * 4) : "memory");
    };
#define ARRIVED                                                                                    \
  if (!((ready >> t) & 1)) {                                                                       \
    ready |= __ballot(lds_ld(tabP) >= call);                                                       \
    if (!((ready >> t) & 1)) {                                                                     \
      TIC();                                                                                       \
      const u64 t0 = __builtin_amdgcn_s_memrealtime();                                             \
      for (;;) {                                                                                   \
        poll(t);                                                                                   \
        wait_vm<0>();                                                                              \
        ready |= __ballot(lds_ld(tabP) >= call);                                                   \
        if ((ready >> t) & 1) break;                                                               \
                                \
        if (give_up(tmo, call, t0, lane)) { rs[2] = 0; ready = ~0ull; break; }                     \
        __builtin_amdgcn_s_sleep(POLL_SLEEP);                                                      \
      }                                                                                            \
      TOC(c2);                                                                                     \
    }                                                                                              \
    FENCE();                                                                                       \
  }
#define AHEAD if (!((ready >> (t + POLL_AHEAD)) & 1)) poll(t + POLL_AHEAD);
#pragma unroll 1
    for (; 2 * t + k < NU1; ++t) {
      HEAD
      ARRIVED
      set_m0_5(slot);
      glds<0, true>(va, rs, so); glds<1024, true>(va, rs, so); glds<2048, true>(va, rs, so); glds<3072, true>(va, rs, so);
      MID
      set_m0(slot + 4096);
      glds<0, true>(va, rs, so + 4096); glds<1024, true>(va, rs, so + 4096); glds<2048, true>(va, rs, so + 4096); glds<3072, true>(va, rs, so + 4096);
      NEXT
      AHEAD
      if (t == 4) STAMP(2);
    }
    STAMP(3);
#pragma unroll 1
    for (; 2 * t + k < NU; ++t) {
      HEAD
      ARRIVED
      set_m0_5(slot + kh * 2048);
      glds<0, true>(va, rs, so); glds<1024, true>(va, rs, so);
      MID
      set_m0(slot + kh * 2048 + 4096);
      glds<0, true>(va, rs, so + 4096); glds<1024, true>(va, rs, so + 4096);
      NEXT
      AHEAD
    }
#undef AHEAD
#undef HEAD
#undef MID
#undef NEXT
#undef ARRIVED
    STAMP(4);
    STAMP_OUT(3);
    if (k == 0) RAW_OUT(10, c1, c2, 0, 0);
    return;
  }

  __builtin_amdgcn_s_setprio(2);
  const unsigned w = wave - 7, wm = w >> 1, wn = w & 1;
  unsigned c0;
  asm volatile("global_load_dword %0, %1, off sc1" : "=v"(c0) : "v"(cnt) : "memory");
  Cons c;
  {
    const unsigned r = lane & 15, g = lane >> 4, q = (lane >> 2) & 3, pp = lane & 3;
    c.la[0] = wm * 4096 + kh * 2048 + r * 64 + ((g ^ XSWZ(r)) << 4);
    c.la[1] = c.la[0] ^ 2048;
    const unsigned b0 = L_W + (8 * g + q) * 256 + (((wn * 8 + 2 * pp) ^ (q | ((g & 1) << 3))) << 4) + ((pp & 1) << 3);
    c.lb[0] = b0; c.lb[1] = b0 ^ 8; c.lb[2] = b0 ^ 16; c.lb[3] = b0 ^ 24;
    c.rq = seq_rsrc();
    c.spin = 0;
    c.tv = 0; c.tn = 0;
  }
#pragma unroll
  for (int i = 0; i < 4; ++i)
#pragma unroll
    for (int n = 0; n < 4; ++n) {
      c.acc[i][n] = f32x4{0.f, 0.f, 0.f, 0.f};
      asm volatile("" : "+v"(c.acc[i][n]));
    }
  asm volatile("s_barrier" ::: "memory");
  set_m0(L_FREE + 256 * w);
  c.vfull = L_FULL;
  PIN(c.vfull);
  c.ok = 0;
  need<false>(c);
  STAMP(1);
  c.B[0][0] = rdB(c.lb[0]); c.A[0][0] = rdA(c.la[0], 0); c.B[0][1] = rdB(c.lb[1]); c.A[0][1] = rdA(c.la[0], 1024);
  c.B[0][2] = rdB(c.lb[2]); c.A[0][2] = rdA(c.la[1], 0); c.B[0][3] = rdB(c.lb[3]); c.A[0][3] = rdA(c.la[1], 1024);
  c.vfull += 8;
  c.ok = 0;
  c.nfree = 4;
  c.sa = SLOT;
  c.sw = SLOT;
  u64 ck0 = 0;
#pragma unroll 1
  for (int it = 0; it < (int)(NU1 - 2) / 2; ++it) {
    TRACE();
    unit1<0, 0>(c);
    unit1<1, 0>(c);
  }
  TRACE();
  constexpr int P2 = NU1 & 1;
  if constexpr (P2) unit1<0, 0>(c);
  unit1<P2, 1>(c);
  unit1<P2 ^ 1, 2>(c);
  MFMA_DONE();
  STAMP(3);
  const unsigned spin1 = c.spin;
  {
    const __amdgpu_buffer_rsrc_t rx = __builtin_amdgcn_make_buffer_rsrc(myws + WS_XBUF + b * 32768 + w * 8192, 0, 8192, 0x00020000);
#pragma unroll
    for (int i = 0; i < 2; ++i)
#pragma unroll
      for (int n = 0; n < 4; ++n)
        __builtin_amdgcn_raw_buffer_store_b128(__builtin_bit_cast(u32x4, c.acc[2 + i][n]), rx, lane * 16 + (i * 4 + n) * 1024, 0, XAUX);
    SB();
#pragma unroll
    for (int n = 0; n < 4; ++n) c.B2[P2][n] = rdB(c.lb[n] + c.sw);
    c.sa = nxa(c.sa);
    c.sw = nxw(c.sw);
    c.nfree += 4;
    c.ok = rfl(min4(lds_ld4(c.vfull)));
    SB();
  }
  const unsigned xl = 2 * w * SLOT + (kh ^ 1) * 2048;
  unsigned call;
  auto xstep = [&](auto I) {
    constexpr int i = decltype(I)::value;
    if constexpr (i == XFLAG_AT) {
      asm volatile("s_waitcnt vmcnt(%1)" : "+v"(c0) : "i"(i < (int)NP2 ? i : (int)NP2 - 1) : "memory");
      call = rfl(c0) + 1;
      if (lane == 0) __hip_atomic_store((GLB unsigned*)(myws + WS_XFLAG + b * 128) + w, call, __ATOMIC_RELAXED, __HIP_MEMORY_SCOPE_SYSTEM);
    }
    if constexpr (i == XLOAD_AT) {
      u32x4 rp = mk_rsrc(myws + WS_XBUF + (b ^ 4) * 32768 + w * 8192, 8192);
      const u64 t0 = __builtin_amdgcn_s_memrealtime();
      GLB unsigned* xf = (GLB unsigned*)(myws + WS_XFLAG + (b ^ 4) * 128) + w;
      while (rfl(__hip_atomic_load(xf, __ATOMIC_RELAXED, __HIP_MEMORY_SCOPE_SYSTEM)) < call) {
        if (give_up(tmo, call, t0, lane)) { rp[2] = 0; break; }
        __builtin_amdgcn_s_sleep(1);
      }
      rp[2] = rfl(rp[2]);
      while (__any(lds_ld(L_FREE + (lane & 3) * 256) < NU1)) {}
      FENCE();
#pragma unroll
      for (int j = 0; j < 4; ++j) {
        set_m0(xl + (j >> 1) * SLOT + (j & 1) * 4096);
        glds<0, true>(lane * 16, rp, j * 2048);
        glds<1024, true>(lane * 16, rp, j * 2048);
      }
      set_m0(L_FREE + 256 * w);
      STAMP(4);
    }
    if constexpr (i < (int)NP2) { TRACE(); unit2<P2 ^ (i & 1), i + 1 < (int)NP2>(c); }
  };
  static_assert(XFLAG_AT <= XLOAD_AT && XLOAD_AT <= NP2 && NSA >= 8, "");
  static_for<NP2 + 1>(xstep);
  MFMA_DONE();
  STAMP(5);
  wait_vm<(XLOAD_AT < NP2 ? NP2 - 1 - XLOAD_AT : 0)>();
  float mx = 0.f;
#pragma unroll
  for (int i = 0; i < 2; ++i)
#pragma unroll
    for (int n = 0; n < 4; ++n) {
      c.acc[i][n] += *(LDSP f32x4*)(size_t)(xl + i * SLOT + (n >> 1) * 4096 + (n & 1) * 1024 + lane * 16);
#pragma unroll
      for (int e = 0; e < 4; ++e) {
        c.acc[i][n][e] = fmaxf(c.acc[i][n][e], 0.f);
        mx = fmaxf(mx, c.acc[i][n][e]);
      }
    }
  lds_st(L_WMAX + 4 * w, wave_max_u32(__builtin_bit_cast(unsigned, mx)) + 1);
  u32x2 res;
  do { res = lds_ld2(L_RES); } while (rfl(res[0]) == 0);
  STAMP(6);
  {
    const bool bad = res[0] != 1;
    const unsigned abits = res[1];
    const float amax = __builtin_bit_cast(float, abits);
    const float s = amax / 448.0f;
    const float inv = 1.0f / s;
    const bool odd = (lane >> 4) & 1;
#pragma unroll
    for (int i = 0; i < 2; ++i) {
      unsigned q4[4];
#pragma unroll
      for (int n = 0; n < 4; ++n) {
        f32x4 y = c.acc[i][n];
        int q = __builtin_amdgcn_cvt_pk_fp8_f32(fminf(y[0] * inv, 448.f), fminf(y[1] * inv, 448.f), 0, false);
        q = __builtin_amdgcn_cvt_pk_fp8_f32(fminf(y[2] * inv, 448.f), fminf(y[3] * inv, 448.f), q, true);
        q4[n] = bad ? 0x7f7f7f7fu : (unsigned)q;
      }
      const u32x4 o = {odd ? q4[1] : q4[0], odd ? q4[0] : q4[1], odd ? q4[3] : q4[2], odd ? q4[2] : q4[3]};
      GLB char* op = (GLB char*)p.out + (size_t)(mb * 128 + wm * 64 + (2 * kh + i) * 16 + (lane & 15)) * NDIM + ct * 128 + wn * 64 + (lane >> 4) * 16;
      *(GLB u32x4*)op = o;
    }
  }
  STAMP(7);
  STAMP_OUT(5 + w);
}

#define HIP_CHECK(expr) do { hipError_t _e = (expr); if (_e != hipSuccess) { \
    fprintf(stderr, "HIP error %s at %s:%d\n", hipGetErrorString(_e), __FILE__, __LINE__); \
    return; } } while (0)

extern "C" void kernel_launch(int rank, int nranks, ncclComm_t comm, hipStream_t stream, void* const* d_in,
                              const int* in_sizes, int n_in, void* d_out, int out_size, const int* in_dtypes,
                              int out_dtype, void* const* d_out_all, void* const* d_ws_all, size_t ws_bytes) {
  if (nranks != NRANK || (unsigned)rank >= (unsigned)NRANK || ws_bytes < WS_NEED || n_in < 2 ||
      in_sizes[0] != 4096 * 512 || in_sizes[1] != 4096 * 4096 || out_size != 512 * 4096) {
    fprintf(stderr, "unexpected problem shape\n");
    return;
  }
  HIP_CHECK(hipFuncSetAttribute((const void*)a2a_gemm_kernel, hipFuncAttributeMaxDynamicSharedMemorySize, LDS_BYTES));
  Params p;
  p.x = d_in[0];
  p.w = d_in[1];
  p.out = d_out;
  for (int i = 0; i < NRANK; ++i) p.peer[i] = (char*)d_ws_all[(rank + 1 + i) % NRANK];
  p.rank = rank;
  p.pad = 0;
  hipLaunchKernelGGL(a2a_gemm_kernel, dim3(NWG), dim3(NTHREADS), LDS_BYTES, stream, p);
  HIP_CHECK(hipGetLastError());
}
```

```cpp
#include <hip/hip_runtime.h>
#include <rccl/rccl.h>
#include <cstdio>
#include <cstdint>

typedef unsigned u32x4 __attribute__((ext_vector_type(4)));
typedef unsigned u32x2 __attribute__((ext_vector_type(2)));
typedef __bf16 bf16x4 __attribute__((ext_vector_type(4)));
typedef __bf16 bf16x8 __attribute__((ext_vector_type(8)));
typedef _Float16 f16x2 __attribute__((ext_vector_type(2)));
typedef float f32x4 __attribute__((ext_vector_type(4)));
typedef unsigned long long u64;
#define LDSP __attribute__((address_space(3)))
#define GLB __attribute__((address_space(1)))
#define DI __device__ __forceinline__

constexpr int NRANK = 8;
constexpr int NWG = 256;
constexpr int NTHREADS = 512;
constexpr int NDIM = 4096;
constexpr unsigned XROW = 512 * 2;
constexpr unsigned WROW = NDIM * 2;
constexpr unsigned X_BYTES = 4096u * XROW;
constexpr unsigned W_BYTES = 4096u * WROW;
constexpr unsigned NU = 32;
constexpr unsigned NGRP = 8;
constexpr unsigned PACE = 35, SEND_T0 = 10;
constexpr unsigned CODE_PF = 20480;
#define XSWZ(row) (((row) >> 3) << 1)

constexpr size_t WS_CNT = 0;
constexpr size_t WS_TMO = 4096;
constexpr size_t WS_G2 = 8192;
constexpr size_t WS_FLAG = 16384;
constexpr size_t WS_XFLAG = 32768;
constexpr size_t WS_G1 = 65536;
constexpr size_t WS_STAGE = 1u << 20;
constexpr unsigned STAGE_BYTES = 4u << 20;
constexpr size_t WS_XBUF = WS_STAGE + STAGE_BYTES;
constexpr unsigned XBUF_BYTES = 12u << 20;
constexpr size_t WS_NEED = WS_XBUF + XBUF_BYTES;

constexpr unsigned NS = 6, SA = 8192, SW = 16384;
constexpr unsigned L_W = NS * SA, L_CTRL = L_W + NS * SW;
constexpr unsigned L_FULL = L_CTRL;
constexpr unsigned L_FREE = L_CTRL + 16;
constexpr unsigned L_GO = L_CTRL + 32;
constexpr unsigned L_WMAX = L_CTRL + 48;
constexpr unsigned L_RES = L_CTRL + 64;
constexpr unsigned L_RAW = L_CTRL + 256;
constexpr unsigned CTRL_BYTES = 512;
constexpr unsigned L_JUNK = L_CTRL + CTRL_BYTES;
constexpr unsigned LDS_BYTES = L_JUNK + 256;

constexpr u64 DEADLINE_TICKS = 50000000ull;

struct Params {
  const void* x;
  const void* w;
  void* out;
  char* peer[NRANK];
  int rank;
  int pad;
};

DI unsigned rfl(unsigned v) { return __builtin_amdgcn_readfirstlane(v); }
DI u32x4 mk_rsrc(const void* ptr, unsigned bytes) {
  u64 a = (u64)ptr;
  u32x4 r = {(unsigned)a, (unsigned)(a >> 32) & 0xffffu, bytes, 0x00020000u};
  return r;
}
DI unsigned lds_ld(unsigned a) { return *(volatile LDSP unsigned*)(size_t)a; }
DI u32x2 lds_ld2(unsigned a) { return *(volatile LDSP u32x2*)(size_t)a; }
DI u32x4 lds_ld4(unsigned a) { return *(volatile LDSP u32x4*)(size_t)a; }
DI void lds_st(unsigned a, unsigned v) { *(volatile LDSP unsigned*)(size_t)a = v; }
#define SB() __builtin_amdgcn_sched_barrier(0)
#define FENCE() asm volatile("" ::: "memory")
#define PIN(v) asm volatile("" : "+v"(v))
#define PINS(v) asm volatile("" : "+s"(v))

template <int N> DI void wait_vm() { asm volatile("s_waitcnt vmcnt(%0)" ::"i"(N) : "memory"); }
DI void set_m0(unsigned v) { asm volatile("s_mov_b32 m0, %0\n\ts_nop 0" ::"s"(v) : "memory"); }
DI void set_m0_5(unsigned v, unsigned& so) { asm volatile("s_mov_b32 m0, %1\n\ts_nop 3" : "+s"(so) : "s"(v) : "memory"); }
template <int IMM, bool SYS>
DI void glds(unsigned voff, u32x4 rsrc, unsigned soff) {
  if constexpr (SYS)
    asm volatile("buffer_load_dwordx4 %0, %1, %2 offen offset:%3 sc0 sc1 lds" ::"v"(voff), "s"(rsrc), "s"(soff), "i"(IMM) : "memory");
  else
    asm volatile("buffer_load_dwordx4 %0, %1, %2 offen offset:%3 lds" ::"v"(voff), "s"(rsrc), "s"(soff), "i"(IMM) : "memory");
}
DI u32x4 seq_rsrc() {
  u64 pc;
  asm volatile("s_getpc_b64 %0\n\ts_branch .Lseq%=\n\t.set .Lv%=, 0\n\t.rept 40\n\t.long .Lv%=\n\t.set .Lv%=, .Lv%=+1\n\t.endr\n.Lseq%=:" : "=s"(pc));
  return mk_rsrc((const void*)(pc + 4), 160);
}
DI void count(u32x4 rq, unsigned off) {
  asm volatile("s_mov_b64 exec, 1\n\tbuffer_load_dword off, %0, %1 lds\n\ts_mov_b64 exec, -1" ::"s"(rq), "s"(off) : "memory");
}
DI unsigned wave_max_u32(unsigned v) {
  v = max(v, (unsigned)__builtin_amdgcn_update_dpp(0, v, 0x111, 0xf, 0xf, true));
  v = max(v, (unsigned)__builtin_amdgcn_update_dpp(0, v, 0x112, 0xf, 0xf, true));
  v = max(v, (unsigned)__builtin_amdgcn_update_dpp(0, v, 0x114, 0xf, 0xf, true));
  v = max(v, (unsigned)__builtin_amdgcn_update_dpp(0, v, 0x118, 0xf, 0xf, true));
  unsigned a = __builtin_amdgcn_readlane(v, 15), b = __builtin_amdgcn_readlane(v, 31);
  unsigned c = __builtin_amdgcn_readlane(v, 47), d = __builtin_amdgcn_readlane(v, 63);
  return max(max(a, b), max(c, d));
}
DI void unit_of(unsigned s, unsigned rank, unsigned kq, unsigned rot, unsigned& src, unsigned& f) {
  if (s < 4) { src = rank; f = 4 * kq + (s ^ (rot & 3)); return; }
  const unsigned u = s - 4, j = u / 7;
  unsigned i = u - 7 * j;
  if (j > 0) { f = 4 * j + kq; src = (rank + 1 + (i + rot) % 7) & 7; return; }
  unsigned pi = 0;
  for (f = 0; f < 4; ++f) {
    const unsigned p0 = (kq - f) & 3, n = p0 == 3 ? 1 : 2;
    if (i < n) { pi = p0 + 4 * i; break; }
    i -= n;
  }
  src = (rank + 1 + pi) & 7;
}

DI f32x4 MFMA(bf16x8 a, bf16x8 b, f32x4 c) {
  asm volatile("v_mfma_f32_16x16x32_bf16 %0, %1, %2, %0" : "+v"(c) : "v"(a), "v"(b));
  return c;
}

struct Cons {
  f32x4 acc[8][4];
  bf16x8 A[2][8], B[2][4];
  unsigned la[4], lb[2][4];
  unsigned vfull, fv;
  unsigned need;
  unsigned nfree;
  u32x4 rq;
};
template <int S, int I> DI bf16x8 rdA(Cons& c) { return *(LDSP bf16x8*)(size_t)(c.la[I >> 1] + S * SA + (I & 1) * 1024); }
template <int S, int N, int H> DI bf16x4 rdB(Cons& c) {
  return __builtin_amdgcn_ds_read_tr16_b64_v4bf16((LDSP bf16x4*)(size_t)(c.lb[S / 4][N] + (S % 4) * SW + H * 2048));
}
DI bf16x8 cat(bf16x4 lo, bf16x4 hi) { return __builtin_shufflevector(lo, hi, 0, 1, 2, 3, 4, 5, 6, 7); }
DI void need(Cons& c, u64 missing) {
  if (__builtin_expect(missing != 0, 0)) {
    while (__any(lds_ld(c.vfull) < c.need)) {}
  }
  FENCE();
}
DI float amax4(float m, f32x4 v) {
  asm("v_max3_f32 %0, |%1|, |%2|, %0\n\tv_max3_f32 %0, |%3|, |%4|, %0" : "+v"(m) : "v"(v[0]), "v"(v[1]), "v"(v[2]), "v"(v[3]));
  return m;
}
DI unsigned pk16(float a, float b) {
  unsigned r;
  asm("v_cvt_pk_f16_f32 %0, %1, %2" : "=v"(r) : "v"(a), "v"(b));
  return r;
}
template <int P, int S, int NEXT>
DI void unit(Cons& c) {
  constexpr int Q = P ^ 1;
  bf16x4 lo;
  u64 miss;
#define M(I, N) c.acc[I][N] = MFMA(c.B[P][N], c.A[P][I], c.acc[I][N]); SB();
#define R(X) if constexpr (NEXT != 0) { X; } SB();
  M(0, 0) R((lo = rdB<S, 0, 0>(c)))
  M(0, 1) R((c.B[Q][0] = cat(lo, rdB<S, 0, 1>(c))))
  M(0, 2) R((c.A[Q][0] = rdA<S, 0>(c)))
  M(0, 3) R((lo = rdB<S, 1, 0>(c)))
  M(1, 0) R((c.B[Q][1] = cat(lo, rdB<S, 1, 1>(c))))
  M(1, 1) R((lo = rdB<S, 2, 0>(c)))
  M(1, 2) R((c.B[Q][2] = cat(lo, rdB<S, 2, 1>(c))))
  M(1, 3) R((lo = rdB<S, 3, 0>(c)))
  M(2, 0) R((c.B[Q][3] = cat(lo, rdB<S, 3, 1>(c))))
  M(2, 1) R((c.A[Q][1] = rdA<S, 1>(c)))
  M(2, 2) R((c.A[Q][2] = rdA<S, 2>(c)))
  M(2, 3) R((c.A[Q][3] = rdA<S, 3>(c)))
  M(3, 0) R((c.A[Q][4] = rdA<S, 4>(c)))
  M(3, 1) R((c.A[Q][5] = rdA<S, 5>(c)))
  M(3, 2) R((c.A[Q][6] = rdA<S, 6>(c)))
  M(3, 3) R((c.A[Q][7] = rdA<S, 7>(c)))
  M(4, 0) M(4, 1) M(4, 2) M(4, 3)
  M(5, 0) M(5, 1)
  M(5, 2) if constexpr (NEXT >= 2) {
            asm volatile("" ::"v"(c.A[Q][7]), "v"(c.A[Q][6]), "v"(c.A[Q][5]), "v"(c.A[Q][4]), "v"(c.A[Q][3]), "v"(c.A[Q][2]),
                         "v"(c.A[Q][1]), "v"(c.A[Q][0]), "v"(c.B[Q][0]), "v"(c.B[Q][1]), "v"(c.B[Q][2]), "v"(c.B[Q][3]) : "memory");
            count(c.rq, c.nfree);
          } SB();
  M(5, 3)
  M(6, 0) if constexpr (NEXT >= 2) { c.nfree += 4; c.need += 1; PINS(c.nfree); PINS(c.need); } SB();
  M(6, 1) if constexpr (NEXT == 2) c.fv = lds_ld(c.vfull); SB();
  M(6, 2) M(6, 3)
  M(7, 0) M(7, 1)
  M(7, 2) if constexpr (NEXT == 2) { miss = __ballot(c.fv < c.need); PINS(miss); } SB();
  M(7, 3) if constexpr (NEXT == 2) need(c, miss); SB();
#undef R
#undef M
}
DI void unit_last(Cons& c, __amdgpu_buffer_rsrc_t rx, unsigned vo) {
  u32x4 o;
#define M(I, N) c.acc[I][N] = MFMA(c.B[1][N], c.A[1][I], c.acc[I][N]); SB();
#define C0(T, N) o[0] = pk16(c.acc[T][N][0], c.acc[T][N][1]); o[1] = pk16(c.acc[T][N][2], c.acc[T][N][3]); PIN(o); SB();
#define C1(T, N) o[2] = pk16(c.acc[T][N][0], c.acc[T][N][1]); o[3] = pk16(c.acc[T][N][2], c.acc[T][N][3]); \
                 __builtin_amdgcn_raw_buffer_store_b128(o, rx, vo + ((T - 2) * 2 + (N >> 1)) * 1024, 0, 17); SB();
#define ROW(I, T) M(I, 0) C0(T, 0) M(I, 1) C1(T, 1) M(I, 2) C0(T, 2) M(I, 3) C1(T, 3)
  M(2, 0) M(2, 1) M(2, 2) M(2, 3)
  M(3, 0) asm volatile("s_nop 0"); M(3, 1) asm volatile("s_nop 0"); M(3, 2) asm volatile("s_nop 0"); M(3, 3) asm volatile("s_nop 0"); SB();
  ROW(4, 2) ROW(5, 3) ROW(6, 4) ROW(7, 5) ROW(0, 6) ROW(1, 7)
  asm volatile("s_nop 7\n\ts_nop 1" ::: "memory");
  SB();
#undef ROW
#undef C1
#undef C0
#undef M
}

template <int I> struct ic { static constexpr int value = I; };
template <int N, int I = 0, class F> DI void static_for(F& f) {
  if constexpr (I < N) { f(ic<I>{}); static_for<N, I + 1>(f); }
}

__global__ void __launch_bounds__(NTHREADS, 2) a2a_gemm_kernel(Params p) {
  u64 pc0;
  asm volatile("s_getpc_b64 %0" : "=s"(pc0));
  const unsigned tid = threadIdx.x, lane = tid & 63, wave = rfl(tid >> 6);
  const unsigned b = blockIdx.x;
  const unsigned rank = p.rank;
  const unsigned kq = (b >> 1) & 3, mb = (b >> 3) & 3, ct = (b & 1) * 8 + (b >> 5);
  char* const myws = p.peer[7];
  char* const pws = p.peer[b % 7];
  const unsigned sp = b / 28;
  asm volatile("" ::"s"(p.x), "s"(p.w), "s"(p.out), "s"(myws), "s"(pws), "s"(rank));
  GLB unsigned* const tmo = (GLB unsigned*)(myws + WS_TMO);
  GLB unsigned* const cnt = (GLB unsigned*)(myws + WS_CNT) + b;

  if (wave == 0) {
    __builtin_amdgcn_s_setprio(3);
    const u64 t0 = __builtin_amdgcn_s_memrealtime();
    const unsigned smb = b / 7 & 3, peer = (rank + 1 + b % 7) & 7;
    u32x4 v[16];
    unsigned c0;
    if (sp >= 8) {
      const u32x4 rc = mk_rsrc((const void*)(pc0 & ~127ull), CODE_PF);
      set_m0(L_JUNK);
#pragma unroll
      for (unsigned i = 0; i < (CODE_PF + 8191) / 8192; ++i)
        asm volatile("buffer_load_dword %0, %1, %2 offen lds" ::"v"(lane * 128), "s"(rc), "s"(i * 8192) : "memory");
    }
    {
      const u32x4 rsX = mk_rsrc(p.x, sp < 8 ? X_BYTES : 0);
      const unsigned vo = (lane >> 3) * XROW + (lane & 7) * 16, so = (peer * 512 + smb * 128) * XROW + (sp & 7) * 128;
#pragma unroll
      for (int i = 0; i < 16; ++i)
        asm volatile("buffer_load_dwordx4 %0, %1, %2, %3 offen" : "=v"(v[i]) : "v"(vo), "s"(rsX), "s"(so + i * 8 * XROW) : "memory");
    }
    asm volatile("global_load_dword %0, %1, off sc1" : "=v"(c0) : "v"(cnt) : "memory");
#pragma unroll
    for (unsigned i = 0; i < CTRL_BYTES / 256; ++i) lds_st(L_CTRL + 256 * i + 4 * lane, i == 0 && lane == 2);
    asm volatile("s_waitcnt lgkmcnt(0)\n\ts_barrier" ::: "memory");
    GLB u64* g2 = (GLB u64*)(p.peer[lane & 7] + WS_G2 + (rank * NGRP + b % NGRP) * 128);
    asm volatile("s_waitcnt vmcnt(0)" : "+v"(v[0]), "+v"(v[1]), "+v"(v[2]), "+v"(v[3]), "+v"(v[4]), "+v"(v[5]), "+v"(v[6]), "+v"(v[7]), "+v"(c0), "+v"(g2)::"memory");
    asm volatile("" : "+v"(v[8]), "+v"(v[9]), "+v"(v[10]), "+v"(v[11]), "+v"(v[12]), "+v"(v[13]), "+v"(v[14]), "+v"(v[15]));
    const unsigned call = rfl(c0) + 1;
    if (sp < 8) {
      GLB char* dst = (GLB char*)pws + WS_STAGE + rank * (512u << 10) + (sp * 2 + ((lane >> 2) & 1)) * 32768 + smb * 8192 + (lane >> 3) * 64;
      GLB char* d0 = dst + (lane & 3) * 16;
      GLB char* d1 = dst + 512 + ((lane & 3) ^ 2) * 16;
      GLB unsigned* flag = (GLB unsigned*)(pws + WS_FLAG + (smb * 8 + sp) * 128) + rank;
      const unsigned due = sp == 0 ? 0 : SEND_T0 + (4 * sp + ((smb + sp) & 3)) * PACE;
      while (__builtin_amdgcn_s_memrealtime() - t0 < due) __builtin_amdgcn_s_sleep(2);
#pragma unroll
      for (int i = 0; i < 16; ++i)
        asm volatile("global_store_dwordx4 %0, %1, off offset:%2 sc0 sc1\n\ts_nop 1" ::"v"((i & 1 ? d1 : d0) + (i >> 3) * 4096), "v"(v[i]), "i"((i >> 1 & 3) * 1024) : "memory");
      wait_vm<0>();
      if (lane == 0) __hip_atomic_store(flag, call, __ATOMIC_RELAXED, __HIP_MEMORY_SCOPE_SYSTEM);
      if (sp == 0) lds_st(L_GO, 1);
    }
    while (rfl(lds_ld(L_FREE)) < NU - 1) __builtin_amdgcn_s_sleep(8);
    u32x4 m4;
    do { m4 = lds_ld4(L_WMAX); } while (rfl(min(min(m4[0], m4[1]), min(m4[2], m4[3]))) == 0);
    const u64 t1 = __builtin_amdgcn_s_memrealtime();
    const unsigned wgmax = max(max(m4[0], m4[1]), max(m4[2], m4[3])) - 1;
    {
      GLB char* g1 = (GLB char*)myws + WS_G1 + (b % NGRP) * 128;
      u32x2 old = {0, 0};
      if (lane == 0)
        asm volatile("global_atomic_umax %1, %2, off offset:4\n\tglobal_atomic_add_x2 %0, %1, %3, off sc0\n\ts_waitcnt vmcnt(0)"
                     : "=&v"(old) : "v"(g1), "v"(wgmax), "v"(1ull) : "memory");
      if (rfl(old[0]) + 1 == call * (NWG / NGRP)) {
        const unsigned m = rfl(old[1]);
        if (lane < 8) __hip_atomic_store(g2, ((u64)call << 32) | m, __ATOMIC_RELAXED, __HIP_MEMORY_SCOPE_SYSTEM);
        if (lane == 0) __hip_atomic_store((GLB unsigned*)g1 + 1, 0u, __ATOMIC_RELAXED, __HIP_MEMORY_SCOPE_AGENT);
      }
    }
    unsigned gm = 0;
    bool dead = false;
    const GLB char* gp = (const GLB char*)myws + WS_G2 + (lane & (8 * NGRP - 1)) * 128;
    for (;;) {
      u32x2 a;
      unsigned tv;
      asm volatile(
          "global_load_dwordx2 %0, %2, off sc0 sc1\n\t"
          "global_load_dword %1, %3, off sc1\n\t"
          "s_waitcnt vmcnt(0)"
          : "=&v"(a), "=&v"(tv)
          : "v"(gp), "v"(tmo)
          : "memory");
      if (rfl(tv) == call) { dead = true; break; }
      if (__all(a[1] == call)) { gm = wave_max_u32(a[0]); break; }
      if (__builtin_amdgcn_s_memrealtime() - t1 > DEADLINE_TICKS) {
        if (lane == 0) __hip_atomic_store(tmo, call, __ATOMIC_RELAXED, __HIP_MEMORY_SCOPE_AGENT);
        dead = true;
        break;
      }
    }
    if (lane == 0) {
      *(volatile LDSP u32x2*)(size_t)L_RES = u32x2{dead ? 2u : 1u, gm};
      __hip_atomic_store(cnt, call, __ATOMIC_RELAXED, __HIP_MEMORY_SCOPE_AGENT);
    }
    return;
  }

  if (wave < 4) {
    const u32x4 rq = seq_rsrc();
    unsigned src, f;
    unit_of(lane & 31, rank, kq, 2 * mb + (b >> 5), src, f);
    const unsigned vfree = L_FREE + (lane & 3) * 4;
    unsigned fr = 0;
#define SPACE(s) if (__builtin_expect(__any(fr + NS <= (s)), 0)) { do { fr = lds_ld(vfree); } while (__any(fr + NS <= (s))); } FENCE();
    if (wave >= 2) {
      const unsigned k = wave - 2;
      const unsigned tabS = ((src * 16 + f) << 18) + 4096;
      const u32x4 rs = mk_rsrc((const char*)p.w - 4096, W_BYTES + 4096);
      const unsigned v0 = (lane >> 4) * WROW + ct * 512 + (((lane & 15) ^ (lane >> 4)) << 4), v1 = v0 ^ 128;
      constexpr unsigned D1 = 256 - 1024, D2 = 4 * WROW - 2048, D3 = 4 * WROW + 256 - 3072;
      unsigned slot = L_W + k * SW;
      asm volatile("s_barrier" ::: "memory");
      if (sp == 0) while (rfl(lds_ld(L_GO)) == 0) __builtin_amdgcn_s_sleep(1);
#pragma unroll 1
      for (unsigned s = k; s < NU; s += 2) {
        unsigned so = __builtin_amdgcn_readlane(tabS, s);
        SPACE(s)
        set_m0_5(slot, so);
        glds<0, false>(v0, rs, so); glds<1024, false>(v0, rs, so + D1); glds<2048, false>(v0, rs, so + D2); glds<3072, false>(v0, rs, so + D3);
        set_m0(slot + 4096);
        glds<0, false>(v1, rs, so + 8 * WROW); glds<1024, false>(v1, rs, so + 8 * WROW + D1);
        glds<2048, false>(v1, rs, so + 8 * WROW + D2); glds<3072, false>(v1, rs, so + 8 * WROW + D3);
        set_m0(slot + 8192);
        glds<0, false>(v0, rs, so + 16 * WROW); glds<1024, false>(v0, rs, so + 16 * WROW + D1);
        glds<2048, false>(v0, rs, so + 16 * WROW + D2); glds<3072, false>(v0, rs, so + 16 * WROW + D3);
        fr = lds_ld(vfree);
        set_m0(slot + 12288);
        glds<0, false>(v1, rs, so + 24 * WROW); glds<1024, false>(v1, rs, so + 24 * WROW + D1);
        glds<2048, false>(v1, rs, so + 24 * WROW + D2); glds<3072, false>(v1, rs, so + 24 * WROW + D3);
        set_m0(L_FULL + 4 + 4 * k);
        count(rq, 4 * s + 8);
        slot += 2 * SW; if (slot >= L_CTRL) slot -= NS * SW;
      }
      return;
    }
    __builtin_amdgcn_s_setprio(1);
    const unsigned tabS = lane < 4 ? (rank * 512 + mb * 128) * XROW + f * 64 : (src * 16 + f) << 15;
    const unsigned tabP = L_RAW + (f >> 1) * 32 + src * 4;
    unsigned c0;
    asm volatile("global_load_dword %0, %1, off sc1" : "=v"(c0) : "v"(cnt) : "memory");
    u32x4 rs = mk_rsrc((const char*)p.x - 4096, X_BYTES + 4096);
    const unsigned row = lane >> 2;
    const unsigned vx = row * XROW + (((lane & 3) ^ XSWZ(row)) << 4) + 4096, va = lane * 16;
    unsigned slot = 0, s = 0;
    asm volatile("s_barrier" ::: "memory");
    if (sp == 0) while (rfl(lds_ld(L_GO)) == 0) __builtin_amdgcn_s_sleep(1);
#pragma unroll 1
    for (; s < 4; ++s) {
      unsigned so = __builtin_amdgcn_readlane(tabS, s);
      set_m0_5(slot, so);
      glds<0, false>(vx, rs, so); glds<1024, false>(vx, rs, so + 15360);
      glds<2048, false>(vx, rs, so + 2 * 15360); glds<3072, false>(vx, rs, so + 3 * 15360);
      set_m0(slot + 4096);
      glds<0, false>(vx, rs, so + 65536); glds<1024, false>(vx, rs, so + 65536 + 15360);
      glds<2048, false>(vx, rs, so + 65536 + 2 * 15360); glds<3072, false>(vx, rs, so + 65536 + 3 * 15360);
      set_m0(L_FULL);
      count(rq, 4 * s + 4);
      slot += SA;
    }
    asm volatile("s_waitcnt vmcnt(36)" : "+v"(c0)::"memory");
    const unsigned call = rfl(c0) + 1;
    rs = mk_rsrc(myws + WS_STAGE + mb * 8192, STAGE_BYTES - mb * 8192);
    u64 ready = ~0ull << NU;
    const u32x4 rsF = mk_rsrc(myws + WS_FLAG + mb * 1024, 1024);
    const unsigned vlane = (lane >> 3) * 128 + (lane & 7) * 4;
    auto poll = [&](unsigned i, bool two) {
      const unsigned a = __builtin_amdgcn_readlane(tabP, i) & (two ? ~63u : ~31u);
      const u64 ex = two ? 0xffffull : 0xffull;
      asm volatile("s_mov_b32 m0, %2\n\ts_mov_b64 exec, %4\n\ts_nop 2\n\tbuffer_load_dword %0, %1, %3 offen sc0 sc1 lds\n\ts_mov_b64 exec, -1"
                   ::"v"(vlane), "s"(rsF), "s"(a), "s"((a - L_RAW) * 4), "s"(ex) : "memory");
    };
    auto look = [&]() { ready |= __ballot(lds_ld(tabP) >= call) & 0xffffffffull; };
#pragma unroll 1
    for (; s < NU; ++s) {
      unsigned so = __builtin_amdgcn_readlane(tabS, s);
      SPACE(s)
      if (~ready >> s & 7) look();
      if (!((ready >> s) & 1)) {
        const u64 t0 = __builtin_amdgcn_s_memrealtime();
        wait_vm<10>();
        look();
        for (unsigned n = 1; !((ready >> s) & 1); ++n) {
          poll(s, false);
          wait_vm<0>();
          look();
          if ((ready >> s) & 1) break;
          if (n % 16 == 0 && (rfl(__hip_atomic_load(tmo, __ATOMIC_RELAXED, __HIP_MEMORY_SCOPE_AGENT)) == call ||
                              __builtin_amdgcn_s_memrealtime() - t0 > DEADLINE_TICKS)) {
            if (lane == 0) __hip_atomic_store(tmo, call, __ATOMIC_RELAXED, __HIP_MEMORY_SCOPE_AGENT);
            rs[2] = 0;
            ready = ~0ull;
            break;
          }
          __builtin_amdgcn_s_sleep(2);
        }
      }
      FENCE();
      set_m0_5(slot, so);
      glds<0, true>(va, rs, so); glds<1024, true>(va, rs, so); glds<2048, true>(va, rs, so); glds<3072, true>(va, rs, so);
      fr = lds_ld(vfree);
      set_m0(slot + 4096);
      glds<0, true>(va, rs, so + 4096); glds<1024, true>(va, rs, so + 4096); glds<2048, true>(va, rs, so + 4096); glds<3072, true>(va, rs, so + 4096);
      set_m0(L_FULL);
      count(rq, 4 * s + 4);
      slot += SA; if (slot == L_W) slot = 0;
      const u64 ahead = ~ready >> (s + 1) & 3;
      if (ahead) { const unsigned sa = s + 1 + __builtin_ctzll(ahead); poll(sa, sa < 11); }
    }
#undef SPACE
    return;
  }

  __builtin_amdgcn_s_setprio(2);
  const unsigned w = wave - 4;
  unsigned c0;
  asm volatile("global_load_dword %0, %1, off sc1" : "=v"(c0) : "v"(cnt) : "memory");
  Cons c;
  {
    const unsigned r = lane & 15, g = lane >> 4, q = (lane >> 2) & 3, pp = lane & 3;
#pragma unroll
    for (int i = 0; i < 4; ++i) { c.la[i] = ((i + kq) & 3) * 2048 + r * 64 + ((g ^ XSWZ(r)) << 4); PIN(c.la[i]); }
    const unsigned b0 = L_W + g * 4096 + (w >> 1) * 1024 + q * 256 + ((((w & 1) * 8 + 2 * pp) ^ (q | ((g & 1) << 3))) << 4) + ((pp & 1) << 3);
#pragma unroll
    for (int n = 0; n < 4; ++n) { c.lb[0][n] = b0 ^ (8 * n); c.lb[1][n] = c.lb[0][n] + 4 * SW; PIN(c.lb[0][n]); PIN(c.lb[1][n]); }
    c.rq = seq_rsrc();
  }
#pragma unroll
  for (int i = 0; i < 8; ++i)
#pragma unroll
    for (int n = 0; n < 4; ++n) {
      c.acc[i][n] = f32x4{0.f, 0.f, 0.f, 0.f};
      PIN(c.acc[i][n]);
    }
  asm volatile("s_barrier" ::: "memory");
  set_m0(L_FREE + 4 * w);
  c.vfull = L_FULL + (lane & 3) % 3 * 4;
  PIN(c.vfull);
  auto fetch = [&](auto I) {
    constexpr int S = decltype(I)::value;
    c.B[0][0] = cat(rdB<S, 0, 0>(c), rdB<S, 0, 1>(c)); c.A[0][0] = rdA<S, 0>(c);
    c.B[0][1] = cat(rdB<S, 1, 0>(c), rdB<S, 1, 1>(c)); c.B[0][2] = cat(rdB<S, 2, 0>(c), rdB<S, 2, 1>(c));
    c.B[0][3] = cat(rdB<S, 3, 0>(c), rdB<S, 3, 1>(c));
    c.A[0][1] = rdA<S, 1>(c); c.A[0][2] = rdA<S, 2>(c); c.A[0][3] = rdA<S, 3>(c);
    c.A[0][4] = rdA<S, 4>(c); c.A[0][5] = rdA<S, 5>(c); c.A[0][6] = rdA<S, 6>(c); c.A[0][7] = rdA<S, 7>(c);
  };
  c.need = 1;
  need(c, 1);
  fetch(ic<0>{});
  c.need = 2;
  need(c, 1);
  c.nfree = 8;
  unit<0, 1, 2>(c);
  unit<1, 2, 2>(c);
  unit<0, 3, 3>(c);
  unit<1, 0, 0>(c);
  need(c, 1);
  fetch(ic<4>{});
  c.need += 1;
  c.nfree += 4;
  need(c, 1);
  auto body = [&](auto I) { unit<decltype(I)::value & 1, (decltype(I)::value + 5) % NS, 2>(c); };
#pragma unroll 1
  for (int it = 0; it < 4; ++it) static_for<NS>(body);
  static_for<2>(body);
  unit<0, 1, 1>(c);
  const __amdgpu_buffer_rsrc_t rx = __builtin_amdgcn_make_buffer_rsrc(myws + WS_XBUF + (b * 4 + w) * 12288, 0, 12288, 0x00020000);
  unit_last(c, rx, lane * 16);

  float y[32];
  {
    float mx = 0.f;
#pragma unroll
    for (int i = 2; i < 8; ++i)
#pragma unroll
      for (int n = 0; n < 4; ++n) mx = amax4(mx, c.acc[i][n]);
    const unsigned e = max(wave_max_u32(__builtin_bit_cast(unsigned, mx)) >> 23, 16u);
    unsigned down = 127u << 23;
    if (e - 121u > 20u) {
      const float up = __builtin_bit_cast(float, (268u - e) << 23);
      down = (e - 14u) << 23;
#pragma unroll
      for (int j = 0; j < 12; ++j) {
        u32x4 o;
#pragma unroll
        for (int h = 0; h < 4; ++h) {
          const f32x4 a = c.acc[2 + (j >> 1)][2 * (j & 1) + (h >> 1)];
          o[h] = pk16(a[2 * (h & 1)] * up, a[2 * (h & 1) + 1] * up);
        }
        __builtin_amdgcn_raw_buffer_store_b128(o, rx, lane * 16 + j * 1024, 0, 17);
      }
    }
    asm volatile("s_waitcnt vmcnt(0)" : "+v"(c0)::"memory");
    const unsigned call = rfl(c0) + 1;
    if (lane == 0) __hip_atomic_store((GLB u64*)(myws + WS_XFLAG + b * 128) + w, ((u64)call << 32) | down, __ATOMIC_RELAXED, __HIP_MEMORY_SCOPE_SYSTEM);
    const unsigned lq = (lane & 3) + 1;
    unsigned sc[3] = {0, 0, 0};
    GLB u64* xf = lq == 4 ? (GLB u64*)tmo : (GLB u64*)(myws + WS_XFLAG + ((b & ~6u) | (((kq - lq) & 3) << 1)) * 128) + w;
#pragma unroll
    for (int i = 0; i < 32; ++i) y[i] = c.acc[i >> 4][(i >> 2) & 3][i & 3];
    const u64 t0 = __builtin_amdgcn_s_memrealtime();
    const u32x4 rp = mk_rsrc(myws + WS_XBUF, XBUF_BYTES);
    u32x4 in[3][4];
    unsigned asked = 0, added = 0;
    for (;;) {
      u32x2 fl;
      if (asked != 7)
        asm volatile("global_load_dwordx2 %0, %1, off sc0 sc1" : "=v"(fl) : "v"(xf) : "memory");
      asm volatile("s_waitcnt vmcnt(0)" : "+v"(fl), "+v"(in[0][0]), "+v"(in[0][1]), "+v"(in[0][2]), "+v"(in[0][3]), "+v"(in[1][0]), "+v"(in[1][1]),
                   "+v"(in[1][2]), "+v"(in[1][3]), "+v"(in[2][0]), "+v"(in[2][1]), "+v"(in[2][2]), "+v"(in[2][3])::"memory");
      const unsigned had = asked;
      if (asked != 7) {
        const unsigned seen = (unsigned)__ballot(fl[1] == call) & 7u & ~asked;
#pragma unroll
        for (int q = 1; q < 4; ++q)
          if ((seen >> (q - 1)) & 1) {
            const unsigned so = (((b & ~6u) | (((kq - q) & 3) << 1)) * 4 + w) * 12288 + (q - 1) * 4096;
            sc[q - 1] = __builtin_amdgcn_readlane(fl[0], q - 1);
#pragma unroll
            for (int j = 0; j < 4; ++j)
              asm volatile("buffer_load_dwordx4 %0, %1, %2, %3 offen offset:%4 sc0 sc1" : "+v"(in[q - 1][j]) : "v"(lane * 16), "s"(rp), "s"(so), "i"(j * 1024) : "memory");
          }
        asked |= seen;
      }
#pragma unroll
      for (int q = 1; q < 4; ++q)
        if (((had & ~added) >> (q - 1)) & 1) {
          const float s = __builtin_bit_cast(float, sc[q - 1]);
#pragma unroll
          for (int i = 0; i < 32; ++i) {
            const unsigned u = in[q - 1][i >> 3][(i >> 1) & 3];
            const f16x2 h = __builtin_bit_cast(f16x2, u);
            y[i] = __builtin_fmaf((float)h[i & 1], s, y[i]);
          }
        }
      added = had;
      if (added == 7) break;
      if (asked != 7 && (__builtin_amdgcn_readlane(fl[0], 3) == call || __builtin_amdgcn_s_memrealtime() - t0 > DEADLINE_TICKS)) {
        if (lane == 0) __hip_atomic_store(tmo, call, __ATOMIC_RELAXED, __HIP_MEMORY_SCOPE_AGENT);
        wait_vm<0>();
        break;
      }
    }
  }
  float mx = 0.f;
#pragma unroll
  for (int i = 0; i < 32; ++i) {
    y[i] = fmaxf(y[i], 0.f);
    mx = fmaxf(mx, y[i]);
  }
  lds_st(L_WMAX + 4 * w, wave_max_u32(__builtin_bit_cast(unsigned, mx)) + 1);
  u32x2 res;
  do { res = lds_ld2(L_RES); } while (rfl(res[0]) == 0);
  {
    const bool bad = res[0] != 1;
    const unsigned abits = res[1];
    const float inv = 448.0f / __builtin_bit_cast(float, abits);
    const bool odd = (lane >> 4) & 1;
#pragma unroll
    for (int i = 0; i < 2; ++i) {
      unsigned q4[4];
#pragma unroll
      for (int n = 0; n < 4; ++n) {
        const float* z = y + 16 * i + 4 * n;
        int q = __builtin_amdgcn_cvt_pk_fp8_f32(z[0] * inv, z[1] * inv, 0, false);
        q = __builtin_amdgcn_cvt_pk_fp8_f32(z[2] * inv, z[3] * inv, q, true);
        q4[n] = bad ? 0x7f7f7f7fu : (unsigned)q;
      }
      const u32x4 o = {odd ? q4[1] : q4[0], odd ? q4[0] : q4[1], odd ? q4[3] : q4[2], odd ? q4[2] : q4[3]};
      GLB char* op = (GLB char*)p.out + (size_t)(mb * 128 + (2 * kq + i) * 16 + (lane & 15)) * NDIM + ct * 256 + w * 64 + (lane >> 4) * 16;
      *(GLB u32x4*)op = o;
    }
  }
}

#define HIP_CHECK(expr) do { hipError_t _e = (expr); if (_e != hipSuccess) { \
    fprintf(stderr, "HIP error %s at %s:%d\n", hipGetErrorString(_e), __FILE__, __LINE__); \
    return; } } while (0)

extern "C" void kernel_launch(int rank, int nranks, ncclComm_t comm, hipStream_t stream, void* const* d_in,
                              const int* in_sizes, int n_in, void* d_out, int out_size, const int* in_dtypes,
                              int out_dtype, void* const* d_out_all, void* const* d_ws_all, size_t ws_bytes) {
  if (nranks != NRANK || (unsigned)rank >= (unsigned)NRANK || ws_bytes < WS_NEED || n_in < 2 ||
      in_sizes[0] != 4096 * 512 || in_sizes[1] != 4096 * 4096 || out_size != 512 * 4096) {
    fprintf(stderr, "unexpected problem shape\n");
    return;
  }
  HIP_CHECK(hipFuncSetAttribute((const void*)a2a_gemm_kernel, hipFuncAttributeMaxDynamicSharedMemorySize, LDS_BYTES));
  Params p;
  p.x = d_in[0];
  p.w = d_in[1];
  p.out = d_out;
  for (int i = 0; i < NRANK; ++i) p.peer[i] = (char*)d_ws_all[(rank + 1 + i) % NRANK];
  p.rank = rank;
  p.pad = 0;
  hipLaunchKernelGGL(a2a_gemm_kernel, dim3(NWG), dim3(NTHREADS), LDS_BYTES, stream, p);
  HIP_CHECK(hipGetLastError());
}
```
